```python
import numpy as np

kernel_name = "gpu_a2a_gemm_m4096_k4096_n16384_bf16_gelu"


ABSMAX_RTOL = 0.008

COMPUTE_DTYPE = "bfloat16"
EPILOGUE = "gelu"
IN_QUANT = None
OUT_QUANT = None
IN_DTYPES = {"x": "bfloat16", "w_mat": "bfloat16"}
WORLD_SIZE = 8

def _to_bf16(x):
    u = np.ascontiguousarray(x, dtype=np.float32).view(np.uint32)
    return ((u + 0x7FFF + ((u >> 16) & 1)) & 0xFFFF0000).view(np.float32)

def setup_inputs():

    x = _to_bf16(np.random.randn(4096, 4096).astype(np.float32))
    w_mat = _to_bf16((np.random.randn(4096, 16384) / np.sqrt(4096)).astype(np.float32))
    return {"x": x, "w_mat": w_mat}

def _epilogue(y):
    c = 0.7978845608028654
    return (0.5 * y * (1.0 + np.tanh(c * (y + 0.044715 * y ** 3)))).astype(np.float32)

def reference(x, w_mat):
    y = (x @ w_mat).astype(np.float32)
    return _epilogue(y)
```

```cpp
#include <hip/hip_runtime.h>
#include <rccl/rccl.h>
#include <cstdio>
#include <cstdint>

#define HIP_CHECK(expr) do { hipError_t _e = (expr); if (_e != hipSuccess) { \
    fprintf(stderr, "HIP error %s at %s:%d\n", hipGetErrorString(_e), __FILE__, __LINE__); \
    return; } } while (0)

#define LAS __attribute__((address_space(3)))
#define DI __device__ __forceinline__
typedef __bf16 bf16x8 __attribute__((ext_vector_type(8)));
typedef __bf16 bf16x4 __attribute__((ext_vector_type(4)));
typedef __bf16 bf16x2 __attribute__((ext_vector_type(2)));
typedef float f32x4 __attribute__((ext_vector_type(4)));
typedef float f32x2 __attribute__((ext_vector_type(2)));
typedef unsigned u32x4 __attribute__((ext_vector_type(4)));
typedef unsigned u32x2 __attribute__((ext_vector_type(2)));

namespace cfg {
constexpr int NR = 8;
constexpr int ML = 512;
constexpr int KB = 512;
constexpr int KT = 4096, NT = 16384;
constexpr int BM = 256, BN = 128, BK = 64;
constexpr int NKS = KT / BK;
constexpr int NWG = 256;
#define CFG_NW 4
constexpr int NA = 3;
constexpr int NW = CFG_NW;
constexpr int SLOT = 16384;
constexpr int A_OFF = 0;
constexpr int W_OFF = 2 * NA * SLOT;
constexpr int LDS_BYTES = W_OFF + NW * SLOT;
static_assert(LDS_BYTES <= 163840, "LDS");
constexpr size_t WS_CNT = 0;
constexpr size_t WS_TMO = 4096;
constexpr size_t WS_FLAG = 8192;
constexpr size_t WS_DATA = 1u << 20;
constexpr size_t BLK_BYTES = (size_t)ML * KB * 2;
constexpr size_t WS_NEED = WS_DATA + 2 * NR * BLK_BYTES;
constexpr int NSTAGE = 8;
constexpr unsigned long long DEADLINE = 50000000ull;
}

struct Params {
    const char* x;
    const char* w;
    char* out;
    char* ws[cfg::NR];
    int rank;
    int mode;
    unsigned long long* dbg;
};

DI unsigned rfl(unsigned v) { return __builtin_amdgcn_readfirstlane(v); }
#define WAIT_V(n) asm volatile("s_waitcnt vmcnt(%0)" :: "n"(n) : "memory")
#define BAR() do { asm volatile("" ::: "memory"); __builtin_amdgcn_s_barrier(); asm volatile("" ::: "memory"); } while (0)
DI unsigned vm_outstanding() {
    unsigned r = __builtin_amdgcn_s_getreg(7 | (0 << 6) | (31 << 11));
    return (r & 15u) | ((r >> 18) & 48u);
}

DI float gelu1(float y) {
    const float k1 = -2.0f * 0.7978845608028654f * 1.4426950408889634f;
    const float k3 = k1 * 0.044715f;
    float t = y * y;
    float z = y * __builtin_fmaf(t, k3, k1);
    float e = __builtin_amdgcn_exp2f(z);
    return y * __builtin_amdgcn_rcpf(1.0f + e);
}
DI unsigned pk_bf16(float lo, float hi) { f32x2 v = {lo, hi}; bf16x2 b = __builtin_convertvector(v, bf16x2); return __builtin_bit_cast(unsigned, b); }

#define MFMA1(c, a, b) c = __builtin_amdgcn_mfma_f32_16x16x32_bf16(a, b, c, 0, 0, 0)
#define A_REC 131072
#define W_REC (64 * cfg::NT * 2)
#define CFG_AUXW 0
#define CFG_AUXA 0
#define DMA16(rs, dst, vo, so, aux) __builtin_amdgcn_raw_ptr_buffer_load_lds(rs, dst, 16, vo, so, 0, aux)
#define CFG_PC 0
#define CFG_PL 0
#define CFG_BARPOS 27
struct Frags { bf16x8 x[8]; bf16x8 w[4]; };

__global__ __launch_bounds__(512, 2) void a2a_gemm_kernel(Params P) {
    using namespace cfg;
    extern __shared__ __attribute__((aligned(16))) char shm[];
    LAS char* L = (LAS char*)shm;
    const unsigned lbase = (unsigned)(size_t)L;
    const int tid = threadIdx.x, lane = tid & 63;
    const int wave = __builtin_amdgcn_readfirstlane(tid >> 6);
    const int b = blockIdx.x;
    const int xcd = b & 7, bi = b >> 3;
    const int mh = bi & 1;
    const int np = xcd * 16 + (bi >> 1);
    const int rank = P.rank;
    char* const myws = P.ws[rank];
    unsigned* const cnt = (unsigned*)(myws + WS_CNT) + b;
    unsigned* const tmo = (unsigned*)(myws + WS_TMO);
#define END_STAMP()

    if (wave < 4) {
        const int wr = wave >> 1, wc = wave & 1;
        const unsigned call = rfl(__hip_atomic_load(cnt, __ATOMIC_RELAXED, __HIP_MEMORY_SCOPE_AGENT)) + 1u;
        __builtin_amdgcn_s_setprio(CFG_PC);
        const int fr = lane & 15, g = lane >> 4;
        const unsigned xa0 = lbase + A_OFF + wr * NA * SLOT + fr * 128 + ((g ^ (fr >> 1)) << 4);
        const unsigned xa1 = xa0 ^ 64u;
        const int q = (lane & 15) >> 2, pp = lane & 3, gl = q | ((g & 1) << 2);
        unsigned wa[4];
#pragma unroll
        for (int n = 0; n < 4; ++n) wa[n] = lbase + W_OFF + (8 * g + q) * 256 + (((wc * 4 + n) ^ gl) << 5) + pp * 8;

        f32x4 acc[8][4];
#pragma unroll
        for (int m = 0; m < 8; ++m)
#pragma unroll
            for (int n = 0; n < 4; ++n) acc[m][n] = (f32x4){0.f, 0.f, 0.f, 0.f};

        Frags F0, F1;
        unsigned aslot = 0, wslot = 0;
        unsigned ap, wp0, wp1, wp2, wp3;
#define PIN(x) asm volatile("" : "+v"(x))
#define SB() __builtin_amdgcn_sched_barrier(0)
#define LP(a) ((LAS char*)(size_t)(a))
#define TR(a) __builtin_amdgcn_ds_read_tr16_b64_v4bf16((LAS bf16x4*)LP(a))
#define CAT(lo, hi) __builtin_shufflevector(lo, hi, 0, 1, 2, 3, 4, 5, 6, 7)
#define XR(m) (*(LAS bf16x8*)LP(ap + (m) * 2048))
#define MF(FC, i) MFMA1(acc[(i) >> 2][(i) & 3], FC.w[(i) & 3], FC.x[(i) >> 2])
#define HEAD(FC, FN, KH) do { \
            bf16x4 lo_, hi_; \
            MF(FC, 0);  lo_ = TR(wp0 + (KH) * 8192);        SB(); \
            MF(FC, 1);  hi_ = TR(wp0 + (KH) * 8192 + 1024); FN.w[0] = CAT(lo_, hi_); SB(); \
            MF(FC, 2);  FN.x[0] = XR(0); SB(); \
            MF(FC, 3);  lo_ = TR(wp1 + (KH) * 8192);        SB(); \
            MF(FC, 4);  hi_ = TR(wp1 + (KH) * 8192 + 1024); FN.w[1] = CAT(lo_, hi_); SB(); \
            MF(FC, 5);  lo_ = TR(wp2 + (KH) * 8192);        SB(); \
            MF(FC, 6);  hi_ = TR(wp2 + (KH) * 8192 + 1024); FN.w[2] = CAT(lo_, hi_); SB(); \
            MF(FC, 7);  lo_ = TR(wp3 + (KH) * 8192);        SB(); \
            MF(FC, 8);  hi_ = TR(wp3 + (KH) * 8192 + 1024); FN.w[3] = CAT(lo_, hi_); SB(); \
            MF(FC, 9);  FN.x[1] = XR(1); SB(); \
            MF(FC, 10); FN.x[2] = XR(2); SB(); \
            MF(FC, 11); FN.x[3] = XR(3); SB(); \
            MF(FC, 12); FN.x[4] = XR(4); SB(); \
            MF(FC, 13); FN.x[5] = XR(5); SB(); \
            MF(FC, 14); FN.x[6] = XR(6); SB(); \
            MF(FC, 15); FN.x[7] = XR(7); SB(); \
            } while (0)
#define BODY0() do { \
            HEAD(F0, F1, 1); \
            MF(F0, 16); aslot += SLOT; if (aslot == NA * SLOT) aslot = 0; wslot += SLOT; if (wslot == NW * SLOT) wslot = 0; SB(); \
            MF(F0, 17); ap = xa0 + aslot; PIN(ap); wp0 = wa[0] + wslot; PIN(wp0); SB(); \
            MF(F0, 18); wp1 = wa[1] + wslot; PIN(wp1); wp2 = wa[2] + wslot; PIN(wp2); SB(); \
            MF(F0, 19); wp3 = wa[3] + wslot; PIN(wp3); SB(); \
            _Pragma("unroll") for (int i_ = 20; i_ < 32; ++i_) { MF(F0, i_); SB(); \
                if (i_ == CFG_BARPOS) { __builtin_amdgcn_s_waitcnt(0xC07F); asm volatile("" ::: "memory"); __builtin_amdgcn_s_barrier(); asm volatile("" ::: "memory"); SB(); } } \
            } while (0)
#define BODY1() do { \
            HEAD(F1, F0, 0); \
            MF(F1, 16); ap = xa1 + aslot; PIN(ap); SB(); \
            _Pragma("unroll") for (int i_ = 17; i_ < 32; ++i_) { MF(F1, i_); SB(); if (i_ == 29) { __builtin_amdgcn_s_waitcnt(0xC07F); SB(); } } \
            } while (0)

        ap = xa0; wp0 = wa[0]; wp1 = wa[1]; wp2 = wa[2]; wp3 = wa[3];
        BAR();
        F0.w[0] = CAT(TR(wp0), TR(wp0 + 1024)); F0.w[1] = CAT(TR(wp1), TR(wp1 + 1024));
        F0.w[2] = CAT(TR(wp2), TR(wp2 + 1024)); F0.w[3] = CAT(TR(wp3), TR(wp3 + 1024));
#pragma unroll
        for (int m = 0; m < 8; ++m) F0.x[m] = XR(m);
        ap = xa1; PIN(ap);
        __builtin_amdgcn_s_waitcnt(0xC07F);
        SB();
        for (int t = 0; t < NKS - 1; ++t) {
            BODY0();
            BODY1();
        }
        HEAD(F0, F1, 1);
#pragma unroll
        for (int i = 16; i < 32; ++i) MF(F0, i);
#pragma unroll
        for (int i = 0; i < 32; ++i) MF(F1, i);
        __builtin_amdgcn_s_setprio(0);
        END_STAMP();
        const bool bad = rfl(__hip_atomic_load(tmo, __ATOMIC_RELAXED, __HIP_MEMORY_SCOPE_AGENT)) == call;
        char* obase = P.out + (size_t)(mh * BM + wr * 128 + fr) * (NT * 2) + (size_t)(np * BN + wc * 64 + (g & 1) * 16 + (g >> 1) * 8) * 2;
#pragma unroll
        for (int m = 0; m < 8; ++m) {
#pragma unroll
            for (int n = 0; n < 4; n += 2) {
                f32x4 a = acc[m][n], c = acc[m][n + 1];
                unsigned a0 = pk_bf16(gelu1(a[0]), gelu1(a[1])), a1 = pk_bf16(gelu1(a[2]), gelu1(a[3]));
                unsigned c0 = pk_bf16(gelu1(c[0]), gelu1(c[1])), c1 = pk_bf16(gelu1(c[2]), gelu1(c[3]));
                auto r0 = __builtin_amdgcn_permlane16_swap(a0, c0, false, false);
                auto r1 = __builtin_amdgcn_permlane16_swap(a1, c1, false, false);
                u32x4 o = {r0[0], r1[0], r0[1], r1[1]};
                if (bad) o = (u32x4){0x7fc07fc0u, 0x7fc07fc0u, 0x7fc07fc0u, 0x7fc07fc0u};
                *(u32x4*)(obase + (size_t)m * 16 * (NT * 2) + n * 32) = o;
            }
        }
    } else if (wave < 6) {
        const int h = wave - 4;
        __builtin_amdgcn_s_setprio(CFG_PL);
        const unsigned vo0 = (lane >> 3) * 1024 + (((lane & 7) ^ (lane >> 4)) << 4);
        const unsigned vo1 = vo0 ^ 64u;
        const char* own = P.x + (size_t)rank * BLK_BYTES + (size_t)(mh * BM + h * 128) * 1024;
        LAS char* ring = L + A_OFF + h * NA * SLOT;
        int slot = 0, jj = 0, qq = 0;
        auto issue = [&](const char* src) {
            __amdgpu_buffer_rsrc_t rs = __builtin_amdgcn_make_buffer_rsrc((void*)src, 0, A_REC, 0x00020000);
            LAS char* dst = ring + slot * SLOT;
#pragma unroll
            for (int i = 0; i < 16; ++i)
                DMA16(rs, (LAS void*)(dst + i * 1024), (i & 1) ? vo1 : vo0, i * 8192, CFG_AUXA);
            slot = (slot + 1 == NA) ? 0 : slot + 1;
        };
        unsigned cnt_old;
        asm volatile("global_load_dword %0, %1, off sc1" : "=v"(cnt_old) : "v"(cnt) : "memory");
#pragma unroll
        for (int t = 0; t < NA; ++t) issue(own + t * 128);
        asm volatile("s_waitcnt vmcnt(%1)" : "+v"(cnt_old) : "n"(16 * (NA - 1)) : "memory");
        const unsigned call = rfl(cnt_old) + 1u;
        const char* rem = myws + WS_DATA + (size_t)((call & 1u) * NR) * BLK_BYTES + (size_t)(mh * BM + h * 128) * 1024;
        const unsigned base = (call - 1u) * NSTAGE;
        const char* flag_ptr = myws + WS_FLAG + lane * 16;
        const bool mine = (lane >> 3) == rank;
        unsigned arrived = (P.mode & 2) ? NSTAGE : 0u;
        const bool poller = (h == 0);
        u32x4 pv = {0u, 0u, 0u, 0u};
        bool pending = false;
        const unsigned long long t0 = __builtin_amdgcn_s_memrealtime();
        auto eval = [&]() {
            unsigned m = pv.x < pv.y ? pv.x : pv.y, m2 = pv.z < pv.w ? pv.z : pv.w;
            m = m < m2 ? m : m2;
            m = mine ? 0xffffffffu : m;
            unsigned a = arrived;
            while (a < NSTAGE && __all(m >= base + a + 1u)) ++a;
            arrived = a;
        };
        BAR();
        for (int t = 0; t < NKS - 1; ++t) {
            if (t + 2 < NKS) asm volatile("s_waitcnt vmcnt(16)" : "+v"(pv) :: "memory");
            else asm volatile("s_waitcnt vmcnt(0)" : "+v"(pv) :: "memory");
            const int tn = t + NA;
            if (poller) {
                if (pending) { eval(); pending = false; }
                if (tn >= 8 && tn < NKS && (unsigned)jj >= arrived) {
                    for (;;) {
                        asm volatile("global_load_dwordx4 %0, %1, off sc0 sc1\n\ts_waitcnt vmcnt(0)" : "=v"(pv) : "v"(flag_ptr) : "memory");
                        eval();
                        if ((unsigned)jj < arrived) break;
                        if (rfl(__hip_atomic_load(tmo, __ATOMIC_RELAXED, __HIP_MEMORY_SCOPE_AGENT)) == call ||
                            __builtin_amdgcn_s_memrealtime() - t0 > DEADLINE) {
                            if (lane == 0) __hip_atomic_store(tmo, call, __ATOMIC_RELAXED, __HIP_MEMORY_SCOPE_AGENT);
                            WAIT_V(0);
                            arrived = NSTAGE; break;
                        }
                        __builtin_amdgcn_s_sleep(2);
                    }
                }
            }
            BAR();
            if (tn < NKS) {
                if (poller && arrived < NSTAGE) {
                    asm volatile("global_load_dwordx4 %0, %1, off sc0 sc1" : "=v"(pv) : "v"(flag_ptr) : "memory");
                    pending = true;
                }
                const char* src;
                if (tn < 8) src = own + tn * 128;
                else {
                    const int p = (rank + 1 + qq) & 7;
                    src = rem + (size_t)p * BLK_BYTES + jj * 128;
                    if (++qq == 7) { qq = 0; ++jj; }
                }
                issue(src);
            }
        }
        END_STAMP();
    } else if (wave == 6) {
        const int l4 = lane >> 4;
        __builtin_amdgcn_s_setprio(CFG_PL);
        const unsigned vo0 = l4 * (NT * 2) + (((lane & 15) ^ (l4 << 1)) << 4);
        const unsigned vo1 = vo0 ^ 128u;
        const char* wb = P.w + (size_t)np * BN * 2;
        LAS char* ring = L + W_OFF;
        int slot = 0, jj = 0, qq = 0;
        auto issue = [&](int krow) {
            __amdgpu_buffer_rsrc_t rs = __builtin_amdgcn_make_buffer_rsrc((void*)(wb + (size_t)krow * (NT * 2)), 0, W_REC, 0x00020000);
            LAS char* dst = ring + slot * SLOT;
#pragma unroll
            for (int i = 0; i < 16; ++i)
                DMA16(rs, (LAS void*)(dst + i * 1024), (i & 2) ? vo1 : vo0, i * 4 * NT * 2, CFG_AUXW);
            slot = (slot + 1 == NW) ? 0 : slot + 1;
        };
        static_assert(NW == 3 || NW == 4, "vmcnt range");
        issue(rank * KB); issue(rank * KB + 64); issue(rank * KB + 128);
        WAIT_V(32);
        if (NW == 4) issue(rank * KB + 192);
        BAR();
        for (int t = 0; t < NKS - 1; ++t) {
            const int rem_tiles = (NKS - 1) - (t + 1);
            if (rem_tiles >= NW - 2) WAIT_V(16 * (NW - 2));
            else if (rem_tiles == 1) WAIT_V(16);
            else WAIT_V(0);
            BAR();
            const int tn = t + NW;
            if (tn < NKS) {
                int krow;
                if (tn < 8) krow = rank * KB + tn * 64;
                else {
                    const int p = (rank + 1 + qq) & 7;
                    krow = p * KB + jj * 64;
                    if (++qq == 7) { qq = 0; ++jj; }
                }
                issue(krow);
            }
        }
        END_STAMP();
    } else {
        const bool pusher = (b < 224) && !(P.mode & 1);
        const int dest = (rank + 1 + (b >> 5)) & 7, pi = b & 31;
        u32x4 v[16];
        const unsigned voff = (lane >> 3) * 1024 + (lane & 7) * 16;
        const unsigned cnt_old = __hip_atomic_load(cnt, __ATOMIC_RELAXED, __HIP_MEMORY_SCOPE_AGENT);
        if (pusher) {
            const char* src = P.x + (size_t)dest * BLK_BYTES + (size_t)pi * 16384 + voff;
#pragma unroll
            for (int i = 0; i < 16; ++i) v[i] = *(const u32x4*)(src + (i & 1) * 8192 + (i >> 1) * 128);
        }
        __builtin_amdgcn_fence(__ATOMIC_ACQUIRE, "");
        WAIT_V(0);
        const unsigned call = rfl(cnt_old) + 1u;
        char* dws = P.ws[dest];
        if (pusher) {
            __amdgpu_buffer_rsrc_t rs = __builtin_amdgcn_make_buffer_rsrc(
                (void*)(dws + WS_DATA + (size_t)((call & 1u) * NR + rank) * BLK_BYTES + (size_t)pi * 16384), 0, 16384, 0x00020000);
#pragma unroll
            for (int i = 0; i < 16; ++i)
                __builtin_amdgcn_raw_buffer_store_b128(v[i], rs, voff + (i & 1) * 8192 + (i >> 1) * 128, 0, 17);
        }
        BAR();
        if (lane == 0) __hip_atomic_store(cnt, call, __ATOMIC_RELAXED, __HIP_MEMORY_SCOPE_AGENT);
        if (pusher) {
            unsigned* fl = (unsigned*)(dws + WS_FLAG) + rank * 32 + pi;
            const unsigned base = (call - 1u) * NSTAGE;
            unsigned issued = 17u, sent = 0u;
            for (int t = 0; t < NKS - 1; ++t) {
                const bool last = (t == NKS - 2);
                if (last) WAIT_V(0);
                const unsigned done = issued - vm_outstanding();
                unsigned st = done >> 1; st = st < (unsigned)NSTAGE ? st : (unsigned)NSTAGE;
                if (st > sent) {
                    asm volatile("" ::: "memory");
                    if (lane == 0) __hip_atomic_store(fl, base + st, __ATOMIC_RELAXED, __HIP_MEMORY_SCOPE_SYSTEM);
                    sent = st; ++issued;
                }
                if (sent == (unsigned)NSTAGE) break;
                BAR();
            }
        }
    }
}

extern "C" void kernel_launch(int rank, int nranks, ncclComm_t comm,
                              hipStream_t stream,
                              void* const* d_in, const int* in_sizes, int n_in,
                              void* d_out, int out_size,
                              const int* in_dtypes, int out_dtype,
                              void* const* d_out_all,
                              void* const* d_ws_all, size_t ws_bytes) {
    using namespace cfg;
    (void)comm; (void)in_dtypes; (void)out_dtype; (void)d_out_all;
    if (nranks != NR || n_in != 2 || (unsigned)rank >= (unsigned)NR || ws_bytes < WS_NEED ||
        in_sizes[0] != KT * KB || in_sizes[1] != KT * NT || out_size != ML * NT) {
        fprintf(stderr, "kernel_launch: unexpected problem shape\n");
        return;
    }
    static bool attr_set = false;
    if (!attr_set) {
        HIP_CHECK(hipFuncSetAttribute((const void*)a2a_gemm_kernel, hipFuncAttributeMaxDynamicSharedMemorySize, LDS_BYTES));
        attr_set = true;
    }
    Params P;
    P.x = (const char*)d_in[0];
    P.w = (const char*)d_in[1];
    P.out = (char*)d_out;
    for (int p = 0; p < NR; ++p) P.ws[p] = (char*)d_ws_all[p];
    P.rank = rank;
    P.mode = 0;
    P.dbg = nullptr;
    hipLaunchKernelGGL(a2a_gemm_kernel, dim3(NWG), dim3(512), LDS_BYTES, stream, P);
    HIP_CHECK(hipGetLastError());
}
```

```cpp
#include <hip/hip_runtime.h>
#include <rccl/rccl.h>
#include <cstdio>
#include <cstdint>

#define HIP_CHECK(expr) do { hipError_t _e = (expr); if (_e != hipSuccess) { \
    fprintf(stderr, "HIP error %s at %s:%d\n", hipGetErrorString(_e), __FILE__, __LINE__); \
    return; } } while (0)

#define LAS __attribute__((address_space(3)))
#define DI __device__ __forceinline__
typedef __bf16 bf16x8 __attribute__((ext_vector_type(8)));
typedef __bf16 bf16x4 __attribute__((ext_vector_type(4)));
typedef __bf16 bf16x2 __attribute__((ext_vector_type(2)));
typedef float f32x4 __attribute__((ext_vector_type(4)));
typedef float f32x2 __attribute__((ext_vector_type(2)));
typedef unsigned u32x4 __attribute__((ext_vector_type(4)));
typedef unsigned u32x2 __attribute__((ext_vector_type(2)));

namespace cfg {
constexpr int NR = 8;
constexpr int ML = 512;
constexpr int KB = 512;
constexpr int KT = 4096, NT = 16384;
constexpr int BM = 256, BN = 128, BK = 64;
constexpr int NKS = KT / BK;
constexpr int NWG = 256;
constexpr int NA = 3;
constexpr int NW = 3;
constexpr int SLOT = 16384;
constexpr int A_OFF = 0;
constexpr int W_OFF = 2 * NA * SLOT;
constexpr int SB_OFF = W_OFF + NW * SLOT;
constexpr int LDS_BYTES = SB_OFF + 2048;
static_assert(LDS_BYTES <= 163840, "LDS");
constexpr size_t WS_CNT = 0;
constexpr size_t WS_TMO = 4096;
constexpr size_t WS_FLAG = 8192;
constexpr size_t WS_DATA = 1u << 20;
constexpr size_t BLK_BYTES = (size_t)ML * KB * 2;
constexpr size_t WS_NEED = WS_DATA + 2 * NR * BLK_BYTES;
constexpr int NSTAGE = 8;
constexpr unsigned long long DEADLINE = 50000000ull;
}

struct Params {
    const char* x;
    const char* w;
    char* out;
    char* ws[cfg::NR];
    int rank;
    int mode;
    unsigned long long* dbg;
};

DI unsigned rfl(unsigned v) { return __builtin_amdgcn_readfirstlane(v); }
#define WAIT_V(n) asm volatile("s_waitcnt vmcnt(%0)" :: "n"(n) : "memory")
#define BAR() do { asm volatile("" ::: "memory"); __builtin_amdgcn_s_barrier(); asm volatile("" ::: "memory"); } while (0)
DI unsigned vm_outstanding() {
    unsigned r = __builtin_amdgcn_s_getreg(7 | (0 << 6) | (31 << 11));
    return (r & 15u) | ((r >> 18) & 48u);
}

DI float gelu1(float y) {
    const float k1 = -2.0f * 0.7978845608028654f * 1.4426950408889634f;
    const float k3 = k1 * 0.044715f;
    float t = y * y;
    float z = y * __builtin_fmaf(t, k3, k1);
    float e = __builtin_amdgcn_exp2f(z);
    return y * __builtin_amdgcn_rcpf(1.0f + e);
}
DI unsigned gelu2(float y0, float y1) {
    const float k1 = -2.0f * 0.7978845608028654f * 1.4426950408889634f;
    const float k3 = k1 * 0.044715f;
    const f32x2 y = {y0, y1};
    const f32x2 z = y * __builtin_elementwise_fma(y * y, (f32x2){k3, k3}, (f32x2){k1, k1});
    f32x2 e; e.x = __builtin_amdgcn_exp2f(z.x); e.y = __builtin_amdgcn_exp2f(z.y);
    const f32x2 d = e + 1.0f;
    f32x2 r; r.x = __builtin_amdgcn_rcpf(d.x); r.y = __builtin_amdgcn_rcpf(d.y);
    const f32x2 o = y * r;
    bf16x2 bb = __builtin_convertvector(o, bf16x2); return __builtin_bit_cast(unsigned, bb);
}
DI unsigned pk_bf16(float lo, float hi) { f32x2 v = {lo, hi}; bf16x2 b = __builtin_convertvector(v, bf16x2); return __builtin_bit_cast(unsigned, b); }

#define MFMA1(c, a, b) c = __builtin_amdgcn_mfma_f32_16x16x32_bf16(a, b, c, 0, 0, 0)
#define A_REC 131072
#define W_REC (64 * cfg::NT * 2)
#define CFG_AUXW 0
#define CFG_AUXA 0
#define DMA16(rs, dst, vo, so, aux) __builtin_amdgcn_raw_ptr_buffer_load_lds(rs, dst, 16, vo, so, 0, aux)
#define CFG_PC 0
#define CFG_PL 0
#define CFG_OAUX 17
#define CFG_SNAKE
#define CFG_R 16
#define CFG_TS 150
#define CFG_TB 250
#define CFG_LOW 2
#define CFG_PRO 2
#define CFG_BARPOS 27
struct Frags { bf16x8 x[8]; bf16x8 w[4]; };

constexpr unsigned long long OWN_BITS = 0x8080808080808080ull;
DI u32x4 lds_rd128(unsigned addr) { u32x4 v; asm volatile("ds_read_b128 %0, %1\n\ts_waitcnt lgkmcnt(0)" : "=v"(v) : "v"(addr) : "memory"); return v; }
struct Sched {
    unsigned long long avail, taken;
    unsigned need;
    DI void init(unsigned call, bool all) { need = call; taken = OWN_BITS; avail = all ? ~0ull : OWN_BITS; }
    DI bool all() const { return avail == ~0ull; }
    DI void update(u32x4 f) {
        unsigned m = f.x < f.y ? f.x : f.y, m2 = f.z < f.w ? f.z : f.w;
        m = m < m2 ? m : m2;
        avail |= __ballot(m >= need);
    }
    DI unsigned spare() const { return (unsigned)__builtin_popcountll(avail & ~taken); }
    DI unsigned total() const { return (unsigned)__builtin_popcountll(avail) - 8u; }
    DI bool next(int& idx) {
        const unsigned long long c = avail & ~taken;
        if (c == 0ull) { idx = 0; return false; }
        idx = __builtin_ctzll(c);
        taken |= 1ull << idx;
        return true;
    }
};

__global__ __launch_bounds__(512, 2) void a2a_gemm_kernel(Params P) {
    using namespace cfg;
    extern __shared__ __attribute__((aligned(16))) char shm[];
    LAS char* L = (LAS char*)shm;
    const unsigned lbase = (unsigned)(size_t)L;
    const int tid = threadIdx.x, lane = tid & 63;
    const int wave = __builtin_amdgcn_readfirstlane(tid >> 6);
    const int b = blockIdx.x;
    const int xcd = b & 7, bi = b >> 3;
    const int mh = bi & 1;
    const int np = xcd * 16 + (bi >> 1);
    const int rank = P.rank;
    char* const myws = P.ws[rank];
    unsigned* const cnt = (unsigned*)(myws + WS_CNT) + b;
    unsigned* const tmo = (unsigned*)(myws + WS_TMO);
#define END_STAMP()

    if (wave < 4) {
        const int wr = wave >> 1, wc = wave & 1;
        const unsigned call = rfl(__hip_atomic_load(cnt, __ATOMIC_RELAXED, __HIP_MEMORY_SCOPE_AGENT)) + 1u;
        __builtin_amdgcn_s_setprio(CFG_PC);
        const int fr = lane & 15, g = lane >> 4;
        const unsigned xa0 = lbase + A_OFF + wr * NA * SLOT + fr * 128 + ((g ^ (fr >> 1)) << 4);
        const unsigned xa1 = xa0 ^ 64u;
        const int q = (lane & 15) >> 2, pp = lane & 3, gl = q | ((g & 1) << 2);
        unsigned wa[4];
#pragma unroll
        for (int n = 0; n < 4; ++n) wa[n] = lbase + W_OFF + (8 * g + q) * 256 + (((wc * 4 + n) ^ gl) << 5) + pp * 8;

        f32x4 acc[8][4];
#pragma unroll
        for (int m = 0; m < 8; ++m)
#pragma unroll
            for (int n = 0; n < 4; ++n) acc[m][n] = (f32x4){0.f, 0.f, 0.f, 0.f};

        Frags F0, F1;
        unsigned aslot = 0, wslot = 0;
        unsigned ap, wp0, wp1, wp2, wp3;
#define PIN(x) asm volatile("" : "+v"(x))
#define SB() __builtin_amdgcn_sched_barrier(0)
#define LP(a) ((LAS char*)(size_t)(a))
#define TR(a) __builtin_amdgcn_ds_read_tr16_b64_v4bf16((LAS bf16x4*)LP(a))
#define CAT(lo, hi) __builtin_shufflevector(lo, hi, 0, 1, 2, 3, 4, 5, 6, 7)
#define XR(m) (*(LAS bf16x8*)LP(ap + (m) * 2048))
#define NI(i) ((((i) >> 2) & 1) ? 3 - ((i) & 3) : ((i) & 3))
#define MF(FC, i) MFMA1(acc[(i) >> 2][NI(i)], FC.w[NI(i)], FC.x[(i) >> 2])
#define HEAD(FC, FN, KH) do { \
            bf16x4 lo_, hi_; \
            MF(FC, 0);  lo_ = TR(wp0 + (KH) * 8192);        SB(); \
            MF(FC, 1);  hi_ = TR(wp0 + (KH) * 8192 + 1024); FN.w[0] = CAT(lo_, hi_); SB(); \
            MF(FC, 2);  FN.x[0] = XR(0); SB(); \
            MF(FC, 3);  lo_ = TR(wp1 + (KH) * 8192);        SB(); \
            MF(FC, 4);  hi_ = TR(wp1 + (KH) * 8192 + 1024); FN.w[1] = CAT(lo_, hi_); SB(); \
            MF(FC, 5);  lo_ = TR(wp2 + (KH) * 8192);        SB(); \
            MF(FC, 6);  hi_ = TR(wp2 + (KH) * 8192 + 1024); FN.w[2] = CAT(lo_, hi_); SB(); \
            MF(FC, 7);  lo_ = TR(wp3 + (KH) * 8192);        SB(); \
            MF(FC, 8);  hi_ = TR(wp3 + (KH) * 8192 + 1024); FN.w[3] = CAT(lo_, hi_); SB(); \
            MF(FC, 9);  FN.x[1] = XR(1); SB(); \
            MF(FC, 10); FN.x[2] = XR(2); SB(); \
            MF(FC, 11); FN.x[3] = XR(3); SB(); \
            MF(FC, 12); FN.x[4] = XR(4); SB(); \
            MF(FC, 13); FN.x[5] = XR(5); SB(); \
            MF(FC, 14); FN.x[6] = XR(6); SB(); \
            MF(FC, 15); FN.x[7] = XR(7); SB(); \
            } while (0)
#define BODY0() do { \
            HEAD(F0, F1, 1); \
            MF(F0, 16); aslot += SLOT; if (aslot == NA * SLOT) aslot = 0; wslot += SLOT; if (wslot == NW * SLOT) wslot = 0; SB(); \
            MF(F0, 17); ap = xa0 + aslot; PIN(ap); wp0 = wa[0] + wslot; PIN(wp0); SB(); \
            MF(F0, 18); wp1 = wa[1] + wslot; PIN(wp1); wp2 = wa[2] + wslot; PIN(wp2); SB(); \
            MF(F0, 19); wp3 = wa[3] + wslot; PIN(wp3); SB(); \
            _Pragma("unroll") for (int i_ = 20; i_ < 32; ++i_) { MF(F0, i_); SB(); \
                if (i_ == CFG_BARPOS) { __builtin_amdgcn_s_waitcnt(0xC07F); asm volatile("" ::: "memory"); __builtin_amdgcn_s_barrier(); asm volatile("" ::: "memory"); SB(); } } \
            } while (0)
#define BODY1() do { \
            HEAD(F1, F0, 0); \
            MF(F1, 16); ap = xa1 + aslot; PIN(ap); SB(); \
            _Pragma("unroll") for (int i_ = 17; i_ < 32; ++i_) { MF(F1, i_); SB(); if (i_ == 29) { __builtin_amdgcn_s_waitcnt(0xC07F); SB(); } } \
            } while (0)

        ap = xa0; wp0 = wa[0]; wp1 = wa[1]; wp2 = wa[2]; wp3 = wa[3];
        BAR();
        F0.w[0] = CAT(TR(wp0), TR(wp0 + 1024)); F0.w[1] = CAT(TR(wp1), TR(wp1 + 1024));
        F0.w[2] = CAT(TR(wp2), TR(wp2 + 1024)); F0.w[3] = CAT(TR(wp3), TR(wp3 + 1024));
#pragma unroll
        for (int m = 0; m < 8; ++m) F0.x[m] = XR(m);
        ap = xa1; PIN(ap);
        __builtin_amdgcn_s_waitcnt(0xC07F);
        SB();
        for (int t = 0; t < NKS - 1; ++t) {
            BODY0();
            BODY1();
        }
        HEAD(F0, F1, 1);
#pragma unroll
        for (int i = 16; i < 32; ++i) MF(F0, i);
#pragma unroll
        for (int i = 0; i < 32; ++i) MF(F1, i);
        __builtin_amdgcn_s_setprio(0);
        END_STAMP();
        const bool bad = rfl(__hip_atomic_load(tmo, __ATOMIC_RELAXED, __HIP_MEMORY_SCOPE_AGENT)) == call;
        char* obase = P.out + (size_t)(mh * BM + wr * 128 + fr) * (NT * 2) + (size_t)(np * BN + wc * 64 + (g & 1) * 16 + (g >> 1) * 8) * 2;
        const __amdgpu_buffer_rsrc_t ors = __builtin_amdgcn_make_buffer_rsrc((void*)(P.out + (size_t)(mh * BM + wr * 128) * (NT * 2) + (size_t)(np * BN + wc * 64) * 2), 0, 128 * NT * 2, 0x00020000);
        const unsigned ovo = fr * (NT * 2) + ((g & 1) * 16 + (g >> 1) * 8) * 2;
        if (__builtin_expect(bad, 0)) {
#pragma unroll 1
            for (int i = 0; i < 16; ++i)
                *(u32x4*)(obase + (size_t)(i >> 1) * 16 * (NT * 2) + (i & 1) * 64) = (u32x4){0x7fc07fc0u, 0x7fc07fc0u, 0x7fc07fc0u, 0x7fc07fc0u};
        } else {
#pragma unroll
            for (int m = 0; m < 8; ++m) {
#pragma unroll
                for (int n = 0; n < 4; n += 2) {
                    f32x4 a = acc[m][n], c = acc[m][n + 1];
                    unsigned a0 = gelu2(a[0], a[1]), a1 = gelu2(a[2], a[3]);
                    unsigned c0 = gelu2(c[0], c[1]), c1 = gelu2(c[2], c[3]);
                    auto r0 = __builtin_amdgcn_permlane16_swap(a0, c0, false, false);
                    auto r1 = __builtin_amdgcn_permlane16_swap(a1, c1, false, false);
                    u32x4 o = {r0[0], r1[0], r0[1], r1[1]};
                    __builtin_amdgcn_raw_buffer_store_b128(o, ors, ovo + m * 16 * (NT * 2) + n * 32, 0, CFG_OAUX);
                }
            }
        }
    } else if (wave < 6) {
        const int h = wave - 4;
        __builtin_amdgcn_s_setprio(CFG_PL);
        const unsigned vo0 = (lane >> 3) * 1024 + (((lane & 7) ^ (lane >> 4)) << 4);
        const unsigned vo1 = vo0 ^ 64u;
        const char* own = P.x + (size_t)rank * BLK_BYTES + (size_t)(mh * BM + h * 128) * 1024;
        LAS char* ring = L + A_OFF + h * NA * SLOT;
        int slot = 0;
        auto issue_part = [&](const char* src, int i0, int i1) {
            __amdgpu_buffer_rsrc_t rs = __builtin_amdgcn_make_buffer_rsrc((void*)src, 0, A_REC, 0x00020000);
            LAS char* dst = ring + slot * SLOT;
#pragma unroll
            for (int i = 0; i < 16; ++i) if (i >= i0 && i < i1)
                DMA16(rs, (LAS void*)(dst + i * 1024), (i & 1) ? vo1 : vo0, i * 8192, CFG_AUXA);
            if (i1 == 16) slot = (slot + 1 == NA) ? 0 : slot + 1;
        };
        auto issue = [&](const char* src) { issue_part(src, 0, 16); };
        unsigned cnt_old;
#pragma unroll
        for (int t = 0; t < CFG_PRO; ++t) issue(own + t * 128);
        WAIT_V(16 * (CFG_PRO - 1));
        BAR();
#pragma unroll
        for (int t = CFG_PRO; t < NA; ++t) issue(own + t * 128);
        asm volatile("global_load_dword %0, %1, off sc1" : "=v"(cnt_old) : "v"(cnt) : "memory");
        WAIT_V(17); BAR(); issue(own + 3 * 128);
        asm volatile("s_waitcnt vmcnt(16)" : "+v"(cnt_old) :: "memory");
        const unsigned call = rfl(cnt_old) + 1u;
        const char* rem = myws + WS_DATA + (size_t)((call & 1u) * NR) * BLK_BYTES + (size_t)(mh * BM + h * 128) * 1024;
        Sched S; S.init(call, (P.mode & 2) != 0);
        const bool poller = (h == 0);
        const __amdgpu_buffer_rsrc_t frs = __builtin_amdgcn_make_buffer_rsrc((void*)(myws + WS_FLAG + (size_t)(b % CFG_R) * 1024), 0, 1024, 0x00020000);
        const unsigned fvo = ((((rank + 1 + (lane & 7)) & 7) << 3) | (lane >> 3)) * 16;
        const unsigned long long t0 = __builtin_amdgcn_s_memrealtime();
        const char* src = own + 4 * 128;
        bool have = true;
        for (int t = 1; t < NKS - 1; ++t) {
            const int tn = t + NA;
            if (t + 2 < NKS) WAIT_V(16); else WAIT_V(0);
            const unsigned sb = lbase + SB_OFF + (t & 1) * 1024 + lane * 16;
            if (__builtin_expect(poller && !have, 0)) {
                Sched T = S;
                T.update(lds_rd128(sb));
                if (T.total() < (unsigned)(tn - 7)) {
                    for (;;) {
                        __builtin_amdgcn_raw_ptr_buffer_load_lds(frs, (LAS void*)(L + SB_OFF + (t & 1) * 1024), 16, fvo, 0, 0, 17);
                        WAIT_V(0);
                        T.update(lds_rd128(sb));
                        if (T.total() >= (unsigned)(tn - 7)) break;
                        if (rfl(__hip_atomic_load(tmo, __ATOMIC_RELAXED, __HIP_MEMORY_SCOPE_AGENT)) == call ||
                            __builtin_amdgcn_s_memrealtime() - t0 > DEADLINE) {
                            if (lane == 0) __hip_atomic_store(tmo, call, __ATOMIC_RELAXED, __HIP_MEMORY_SCOPE_AGENT);
                            const u32x4 full = {S.need, S.need, S.need, S.need};
                            asm volatile("ds_write_b128 %0, %1\n\ts_waitcnt vmcnt(0) lgkmcnt(0)" :: "v"(sb), "v"(full) : "memory");
                            break;
                        }
                        __builtin_amdgcn_s_sleep(8);
                    }
                }
            }
            BAR();
            if (tn < NKS) {
                if (__builtin_expect(!have, 0)) {
                    int u;
                    S.update(lds_rd128(sb));
                    S.next(u);
                    src = rem + (size_t)((rank + 1 + u) & 7) * BLK_BYTES + (u >> 3) * 128;
                }
                if (poller && tn + 2 >= 8 && tn + 1 < NKS && !S.all() && S.spare() <= (unsigned)CFG_LOW)
                    __builtin_amdgcn_raw_ptr_buffer_load_lds(frs, (LAS void*)(L + SB_OFF + ((t + 1) & 1) * 1024), 16, fvo, 0, 0, 17);
                issue_part(src, 0, 6);
                const char* nsrc = own + ((tn + 1) & 7) * 128;
                have = true;
                if (tn + 1 >= 8 && tn + 1 < NKS) {
                    int u;
                    if (!S.all()) S.update(lds_rd128(sb));
                    have = S.next(u);
                    nsrc = rem + (size_t)((rank + 1 + u) & 7) * BLK_BYTES + (u >> 3) * 128;
                }
                issue_part(src, 6, 16);
                src = nsrc;
            }
        }
        END_STAMP();
    } else if (wave == 6) {
        const int l4 = lane >> 4;
        __builtin_amdgcn_s_setprio(CFG_PL);
        const unsigned vo0 = l4 * (NT * 2) + (((lane & 15) ^ (l4 << 1)) << 4);
        const unsigned vo1 = vo0 ^ 128u;
        const char* wb = P.w + (size_t)np * BN * 2;
        LAS char* ring = L + W_OFF;
        int slot = 0;
        auto issue_part = [&](int krow, int i0, int i1) {
            __amdgpu_buffer_rsrc_t rs = __builtin_amdgcn_make_buffer_rsrc((void*)(wb + (size_t)krow * (NT * 2)), 0, W_REC, 0x00020000);
            LAS char* dst = ring + slot * SLOT;
#pragma unroll
            for (int i = 0; i < 16; ++i) if (i >= i0 && i < i1)
                DMA16(rs, (LAS void*)(dst + i * 1024), (i & 2) ? vo1 : vo0, i * 4 * NT * 2, CFG_AUXW);
            if (i1 == 16) slot = (slot + 1 == NW) ? 0 : slot + 1;
        };
        auto issue = [&](int krow) { issue_part(krow, 0, 16); };
        unsigned cnt_old;
#pragma unroll
        for (int t = 0; t < CFG_PRO; ++t) issue(rank * KB + t * 64);
        WAIT_V(16 * (CFG_PRO - 1));
        BAR();
#pragma unroll
        for (int t = CFG_PRO; t < NW; ++t) issue(rank * KB + t * 64);
        asm volatile("global_load_dword %0, %1, off sc1" : "=v"(cnt_old) : "v"(cnt) : "memory");
        WAIT_V(17); BAR(); issue(rank * KB + 192);
        asm volatile("s_waitcnt vmcnt(16)" : "+v"(cnt_old) :: "memory");
        const unsigned call = rfl(cnt_old) + 1u;
        Sched S; S.init(call, (P.mode & 2) != 0);
        int krow = rank * KB + 4 * 64;
        bool have = true;
        for (int t = 1; t < NKS - 1; ++t) {
            const int tn = t + NW;
            if (t + 2 < NKS) WAIT_V(16); else WAIT_V(0);
            const unsigned sb = lbase + SB_OFF + (t & 1) * 1024 + lane * 16;
            BAR();
            if (tn < NKS) {
                if (__builtin_expect(!have, 0)) {
                    int u;
                    S.update(lds_rd128(sb));
                    S.next(u);
                    krow = ((rank + 1 + u) & 7) * KB + (u >> 3) * 64;
                }
                issue_part(krow, 0, 6);
                int nk = rank * KB + ((tn + 1) & 7) * 64;
                have = true;
                if (tn + 1 >= 8 && tn + 1 < NKS) {
                    int u;
                    if (!S.all()) S.update(lds_rd128(sb));
                    have = S.next(u);
                    nk = ((rank + 1 + u) & 7) * KB + (u >> 3) * 64;
                }
                issue_part(krow, 6, 16);
                krow = nk;
            }
        }
        END_STAMP();
    } else {
        const bool pusher = (b < 224) && !(P.mode & 1);
        const int dest = (rank + 1 + (b >> 5)) & 7;
        const int quarter = b & 3, stage = (((b & 31) >> 2) + (b >> 5)) & 7;
        u32x4 v[16];
        const unsigned voff = (lane >> 3) * 1024 + (lane & 7) * 16;
        const unsigned long long t0 = __builtin_amdgcn_s_memrealtime();
        const unsigned cnt_old = __hip_atomic_load(cnt, __ATOMIC_RELAXED, __HIP_MEMORY_SCOPE_AGENT);
        if (pusher) {
            const char* src = P.x + (size_t)dest * BLK_BYTES + (size_t)quarter * 131072 + stage * 128 + voff;
#pragma unroll
            for (int i = 0; i < 16; ++i) v[i] = *(const u32x4*)(src + i * 8192);
        }
        asm volatile("buffer_inv sc0 sc1" ::: "memory");
#define CFG_EARLY 1
        int t = -1;
        if (!(pusher && stage < CFG_EARLY))
            while (t < 2 && (t < 0 || vm_outstanding() != 0u)) { BAR(); ++t; }
        WAIT_V(0);
        const unsigned call = rfl(cnt_old) + 1u;
        if (pusher) {
            char* dws = P.ws[dest];
            __amdgpu_buffer_rsrc_t rs = __builtin_amdgcn_make_buffer_rsrc(
                (void*)(dws + WS_DATA + (size_t)((call & 1u) * NR + rank) * BLK_BYTES + (size_t)quarter * 131072 + stage * 128), 0, 131072 - stage * 128, 0x00020000);
            unsigned* fl = (unsigned*)(dws + WS_FLAG) + (rank * 8 + stage) * 4 + quarter;
            const char* myflag = myws + WS_FLAG + (size_t)(b % CFG_R) * 1024 + ((((rank + 1 + (lane & 7)) & 7) << 3) | (lane >> 3)) * 16;
            Sched V; V.init(call, (P.mode & 2) != 0);
            const unsigned long long tgo = t0 + (unsigned long long)(CFG_TB + stage * CFG_TS);
#define CFG_Q 16
            int nst = 0;
            bool go = (stage < CFG_EARLY);
            for (;;) {
                if (!go) go = __builtin_amdgcn_s_memrealtime() >= tgo;
                if (go) {
                    unsigned out_now = vm_outstanding();
                    if (nst == 16 && out_now == 0u) {
                        WAIT_V(0);
                        if (lane < CFG_R) __hip_atomic_store(fl + lane * 256, call, __ATOMIC_RELAXED, __HIP_MEMORY_SCOPE_SYSTEM);
                        break;
                    }
                    while (nst < 16 && out_now + 4u <= (unsigned)CFG_Q) {
#define PST(i) case i: _Pragma("unroll") for (int k = 0; k < 4; ++k) __builtin_amdgcn_raw_buffer_store_b128(v[(i) + k], rs, voff, ((i) + k) * 8192, 17); break;
                        switch (nst) { PST(0) PST(4) PST(8) default: PST(12) }
#undef PST
                        out_now += 4u; nst += 4;
                    }
                }
                if (t >= NKS - 1) { __builtin_amdgcn_s_sleep(1); continue; }
                const int tn = t + NA;
                if (tn >= 8 && tn < NKS && V.total() < (unsigned)(tn - 7)) {
                    V.update(lds_rd128(lbase + SB_OFF + ((t - 1) & 1) * 1024 + lane * 16));
                    if (V.total() < (unsigned)(tn - 7)) {
                        u32x4 f;
                        asm volatile("global_load_dwordx4 %0, %1, off sc0 sc1\n\ts_waitcnt vmcnt(0)" : "=v"(f) : "v"(myflag) : "memory");
                        V.update(f);
                        if (V.total() < (unsigned)(tn - 7)) {
                            if (rfl(__hip_atomic_load(tmo, __ATOMIC_RELAXED, __HIP_MEMORY_SCOPE_AGENT)) == call ||
                                __builtin_amdgcn_s_memrealtime() - t0 > DEADLINE) V.avail = ~0ull;
                            WAIT_V(0);
                            __builtin_amdgcn_s_sleep(4);
                            continue;
                        }
                    }
                }
                BAR(); ++t;
            }
        }
        while (t < 3) { BAR(); ++t; }
        if (lane == 0) __hip_atomic_store(cnt, call, __ATOMIC_RELAXED, __HIP_MEMORY_SCOPE_AGENT);
    }
}

extern "C" void kernel_launch(int rank, int nranks, ncclComm_t comm,
                              hipStream_t stream,
                              void* const* d_in, const int* in_sizes, int n_in,
                              void* d_out, int out_size,
                              const int* in_dtypes, int out_dtype,
                              void* const* d_out_all,
                              void* const* d_ws_all, size_t ws_bytes) {
    using namespace cfg;
    (void)comm; (void)in_dtypes; (void)out_dtype; (void)d_out_all;
    if (nranks != NR || n_in != 2 || (unsigned)rank >= (unsigned)NR || ws_bytes < WS_NEED ||
        in_sizes[0] != KT * KB || in_sizes[1] != KT * NT || out_size != ML * NT) {
        fprintf(stderr, "kernel_launch: unexpected problem shape\n");
        return;
    }
    static bool attr_set = false;
    if (!attr_set) {
        HIP_CHECK(hipFuncSetAttribute((const void*)a2a_gemm_kernel, hipFuncAttributeMaxDynamicSharedMemorySize, LDS_BYTES));
        attr_set = true;
    }
    Params P;
    P.x = (const char*)d_in[0];
    P.w = (const char*)d_in[1];
    P.out = (char*)d_out;
    for (int p = 0; p < NR; ++p) P.ws[p] = (char*)d_ws_all[p];
    P.rank = rank;
    P.mode = 0;
    P.dbg = nullptr;
    hipLaunchKernelGGL(a2a_gemm_kernel, dim3(NWG), dim3(512), LDS_BYTES, stream, P);
    HIP_CHECK(hipGetLastError());
}
```

```cpp
#include <hip/hip_runtime.h>
#include <rccl/rccl.h>
#include <cstdio>
#include <cstdint>

#define HIP_CHECK(expr) do { hipError_t _e = (expr); if (_e != hipSuccess) { \
    fprintf(stderr, "HIP error %s at %s:%d\n", hipGetErrorString(_e), __FILE__, __LINE__); \
    return; } } while (0)

#define LAS __attribute__((address_space(3)))
#define DI __device__ __forceinline__
typedef __bf16 bf16x8 __attribute__((ext_vector_type(8)));
typedef __bf16 bf16x4 __attribute__((ext_vector_type(4)));
typedef __bf16 bf16x2 __attribute__((ext_vector_type(2)));
typedef float f32x4 __attribute__((ext_vector_type(4)));
typedef float f32x2 __attribute__((ext_vector_type(2)));
typedef unsigned u32x4 __attribute__((ext_vector_type(4)));
typedef unsigned u32x2 __attribute__((ext_vector_type(2)));

namespace cfg {
constexpr int NR = 8;
constexpr int ML = 512;
constexpr int KB = 512;
constexpr int KT = 4096, NT = 16384;
constexpr int BM = 256, BN = 128, BK = 64;
constexpr int NKS = KT / BK;
constexpr int NWG = 256;
constexpr int NA = 3;
#define CFG_NW 3
constexpr int NW = CFG_NW;
constexpr int SLOT = 16384;
constexpr int A_OFF = 0;
constexpr int W_OFF = 2 * NA * SLOT;
constexpr int SB_OFF = W_OFF + 3 * SLOT;
constexpr int DUMMY_OFF = SB_OFF + 2048;
constexpr int LDS_BYTES = NW == 4 ? 163840 : DUMMY_OFF + 256;
static_assert(LDS_BYTES <= 163840, "LDS");
constexpr size_t WS_CNT = 0;
constexpr size_t WS_TMO = 4096;
constexpr size_t WS_FLAG = 8192;
constexpr size_t WS_DATA = 1u << 20;
constexpr size_t BLK_BYTES = (size_t)ML * KB * 2;
constexpr unsigned UNIT_BYTES = 74 * 1024;
constexpr size_t SRC_BYTES = (size_t)8 * UNIT_BYTES;
constexpr size_t WS_NEED = WS_DATA + 2 * NR * SRC_BYTES;
constexpr int NSTAGE = 8;
constexpr unsigned long long DEADLINE = 150000000ull;
}

__host__ __device__ inline unsigned stage_off(unsigned r) {
    const unsigned g = r / 7u, s = r - 7u * g;
    return g * 1024u + (s + (s >= 3u ? 1u : 0u)) * 128u;
}

struct Params {
    const char* x;
    const char* w;
    char* out;
    char* ws[cfg::NR];
    int rank;
    int mode;
    unsigned long long* dbg;
};

DI unsigned rfl(unsigned v) { return __builtin_amdgcn_readfirstlane(v); }
#define WAIT_V(n) asm volatile("s_waitcnt vmcnt(%0)" :: "n"(n) : "memory")
#define BAR() do { asm volatile("" ::: "memory"); __builtin_amdgcn_s_barrier(); asm volatile("" ::: "memory"); } while (0)
DI unsigned vm_outstanding() {
    unsigned r = __builtin_amdgcn_s_getreg(7 | (0 << 6) | (31 << 11));
    return (r & 15u) | ((r >> 18) & 48u);
}

DI float gelu1(float y) {
    const float k1 = -2.0f * 0.7978845608028654f * 1.4426950408889634f;
    const float k3 = k1 * 0.044715f;
    float t = y * y;
    float z = y * __builtin_fmaf(t, k3, k1);
    float e = __builtin_amdgcn_exp2f(z);
    return y * __builtin_amdgcn_rcpf(1.0f + e);
}
DI unsigned gelu2(float y0, float y1) {
    const float k1 = -2.0f * 0.7978845608028654f * 1.4426950408889634f;
    const float k3 = k1 * 0.044715f;
    const f32x2 y = {y0, y1};
    const f32x2 z = y * __builtin_elementwise_fma(y * y, (f32x2){k3, k3}, (f32x2){k1, k1});
    f32x2 e; e.x = __builtin_amdgcn_exp2f(z.x); e.y = __builtin_amdgcn_exp2f(z.y);
    const f32x2 d = e + 1.0f;
    f32x2 r; r.x = __builtin_amdgcn_rcpf(d.x); r.y = __builtin_amdgcn_rcpf(d.y);
    const f32x2 o = y * r;
    bf16x2 bb = __builtin_convertvector(o, bf16x2); return __builtin_bit_cast(unsigned, bb);
}
DI unsigned pk_bf16(float lo, float hi) { f32x2 v = {lo, hi}; bf16x2 b = __builtin_convertvector(v, bf16x2); return __builtin_bit_cast(unsigned, b); }

#define MFMA1(c, a, b) c = __builtin_amdgcn_mfma_f32_16x16x32_bf16(a, b, c, 0, 0, 0)
#define A_REC 131072
#define W_REC (63 * cfg::NT * 2 + 512)
#define CFG_AUXW 0
#define CFG_AUXA 0
#define DMA16(rs, dst, vo, so, aux) __builtin_amdgcn_raw_ptr_buffer_load_lds(rs, dst, 16, vo, so, 0, aux)
#define CFG_PC 0
#define CFG_PL 0
#define CFG_OAUX 17
#define CFG_SNAKE
#define CFG_R 16
#define CFG_TS 150
#define CFG_TB 250
#define CFG_LOW 2
#define CFG_PRO 2
#define CFG_BARPOS 27
struct Frags { bf16x8 x[8]; bf16x8 w[4]; };

constexpr unsigned long long OWN_BITS = 0x8080808080808080ull;
DI u32x4 lds_rd128(unsigned addr) { u32x4 v; asm volatile("ds_read_b128 %0, %1\n\ts_waitcnt lgkmcnt(0)" : "=v"(v) : "v"(addr) : "memory"); return v; }
struct Sched {
    unsigned long long avail, taken;
    unsigned need;
    DI void init(unsigned call, bool all) { need = call; taken = OWN_BITS; avail = all ? ~0ull : OWN_BITS; }
    DI bool all() const { return avail == ~0ull; }
    DI void update(u32x4 f) {
        unsigned m = f.x < f.y ? f.x : f.y, m2 = f.z < f.w ? f.z : f.w;
        m = m < m2 ? m : m2;
        avail |= __ballot(m >= need);
    }
    DI unsigned spare() const { return (unsigned)__builtin_popcountll(avail & ~taken); }
    DI unsigned total() const { return (unsigned)__builtin_popcountll(avail) - 8u; }
    DI bool next(int& idx) {
        const unsigned long long c = avail & ~taken;
        if (c == 0ull) { idx = 0; return false; }
        idx = __builtin_ctzll(c);
        taken |= 1ull << idx;
        return true;
    }
};

__global__ __launch_bounds__(512, 2) void a2a_gemm_kernel(Params P) {
    using namespace cfg;
    extern __shared__ __attribute__((aligned(16))) char shm[];
    LAS char* L = (LAS char*)shm;
    const unsigned lbase = (unsigned)(size_t)L;
    const int tid = threadIdx.x, lane = tid & 63;
    const int wave = __builtin_amdgcn_readfirstlane(tid >> 6);
    const int b = blockIdx.x;
    const int xcd = b & 7, bi = b >> 3;
    const int mh = bi & 1;
    const int np = xcd * 16 + (bi >> 1);
    const int ch0 = (np >> 2) * 8 + (np & 2) * 2 + (np & 1), ch1 = ch0 + 2;
    const int rank = P.rank;
    const int rot = xcd, rot7 = xcd == 7 ? 0 : xcd;
    auto src_of = [&](int u) { int q = (u & 7) + rot7; q = q >= 7 ? q - 7 : q; return (u & 7) == 7 ? rank : ((rank + 1 + q) & 7); };
    char* const myws = P.ws[rank];
    unsigned* const cnt = (unsigned*)(myws + WS_CNT) + b;
    unsigned* const tmo = (unsigned*)(myws + WS_TMO);
#define END_STAMP()

    if (wave < 4) {
        const int wr = wave >> 1, wc = wave & 1;
        const unsigned call = rfl(__hip_atomic_load(cnt, __ATOMIC_RELAXED, __HIP_MEMORY_SCOPE_AGENT)) + 1u;
        __builtin_amdgcn_s_setprio(CFG_PC);
        const int fr = lane & 15, g = lane >> 4;
        const unsigned xa0 = lbase + A_OFF + wr * NA * SLOT + fr * 128 + ((g ^ (fr >> 1)) << 4);
        const unsigned xa1 = xa0 ^ 64u;
        const int q = (lane & 15) >> 2, pp = lane & 3, gl = q | ((g & 1) << 2);
        unsigned wa[4];
#pragma unroll
        for (int n = 0; n < 4; ++n) wa[n] = lbase + W_OFF + (8 * g + q) * 256 + (((wc * 4 + n) ^ gl) << 5) + pp * 8;

        f32x4 acc[8][4];
#pragma unroll
        for (int m = 0; m < 8; ++m)
#pragma unroll
            for (int n = 0; n < 4; ++n) acc[m][n] = (f32x4){0.f, 0.f, 0.f, 0.f};

        Frags F0, F1;
        unsigned aslot = 0, wslot = 0;
        unsigned ap, wp0, wp1, wp2, wp3;
#define PIN(x) asm volatile("" : "+v"(x))
#define SB() __builtin_amdgcn_sched_barrier(0)
#define LP(a) ((LAS char*)(size_t)(a))
#define TR(a) __builtin_amdgcn_ds_read_tr16_b64_v4bf16((LAS bf16x4*)LP(a))
#define CAT(lo, hi) __builtin_shufflevector(lo, hi, 0, 1, 2, 3, 4, 5, 6, 7)
#define XR(m) (*(LAS bf16x8*)LP(ap + (m) * 2048))
#define NI(i) ((((i) >> 2) & 1) ? 3 - ((i) & 3) : ((i) & 3))
#define MF(FC, i) MFMA1(acc[(i) >> 2][NI(i)], FC.w[NI(i)], FC.x[(i) >> 2])
#define HEAD(FC, FN, KH) do { \
            bf16x4 lo_, hi_; \
            MF(FC, 0);  lo_ = TR(wp0 + (KH) * 8192);        SB(); \
            MF(FC, 1);  hi_ = TR(wp0 + (KH) * 8192 + 1024); FN.w[0] = CAT(lo_, hi_); SB(); \
            MF(FC, 2);  FN.x[0] = XR(0); SB(); \
            MF(FC, 3);  lo_ = TR(wp1 + (KH) * 8192);        SB(); \
            MF(FC, 4);  hi_ = TR(wp1 + (KH) * 8192 + 1024); FN.w[1] = CAT(lo_, hi_); SB(); \
            MF(FC, 5);  lo_ = TR(wp2 + (KH) * 8192);        SB(); \
            MF(FC, 6);  hi_ = TR(wp2 + (KH) * 8192 + 1024); FN.w[2] = CAT(lo_, hi_); SB(); \
            MF(FC, 7);  lo_ = TR(wp3 + (KH) * 8192);        SB(); \
            MF(FC, 8);  hi_ = TR(wp3 + (KH) * 8192 + 1024); FN.w[3] = CAT(lo_, hi_); SB(); \
            MF(FC, 9);  FN.x[1] = XR(1); SB(); \
            MF(FC, 10); FN.x[2] = XR(2); SB(); \
            MF(FC, 11); FN.x[3] = XR(3); SB(); \
            MF(FC, 12); FN.x[4] = XR(4); SB(); \
            MF(FC, 13); FN.x[5] = XR(5); SB(); \
            MF(FC, 14); FN.x[6] = XR(6); SB(); \
            MF(FC, 15); FN.x[7] = XR(7); SB(); \
            } while (0)
#define BODY0() do { \
            HEAD(F0, F1, 1); \
            MF(F0, 16); aslot += SLOT; if (aslot == NA * SLOT) aslot = 0; wslot += SLOT; if (wslot == NW * SLOT) wslot = 0; SB(); \
            MF(F0, 17); ap = xa0 + aslot; PIN(ap); wp0 = wa[0] + wslot; PIN(wp0); SB(); \
            MF(F0, 18); wp1 = wa[1] + wslot; PIN(wp1); wp2 = wa[2] + wslot; PIN(wp2); SB(); \
            MF(F0, 19); wp3 = wa[3] + wslot; PIN(wp3); SB(); \
            _Pragma("unroll") for (int i_ = 20; i_ < 32; ++i_) { MF(F0, i_); SB(); \
                if (i_ == CFG_BARPOS) { __builtin_amdgcn_s_waitcnt(0xC07F); asm volatile("" ::: "memory"); __builtin_amdgcn_s_barrier(); asm volatile("" ::: "memory"); SB(); } } \
            } while (0)
#define BODY1() do { \
            HEAD(F1, F0, 0); \
            MF(F1, 16); ap = xa1 + aslot; PIN(ap); SB(); \
            _Pragma("unroll") for (int i_ = 17; i_ < 32; ++i_) { MF(F1, i_); SB(); if (i_ == 29) { __builtin_amdgcn_s_waitcnt(0xC07F); SB(); } } \
            } while (0)

        ap = xa0; wp0 = wa[0]; wp1 = wa[1]; wp2 = wa[2]; wp3 = wa[3];
        BAR();
        F0.w[0] = CAT(TR(wp0), TR(wp0 + 1024)); F0.w[1] = CAT(TR(wp1), TR(wp1 + 1024));
        F0.w[2] = CAT(TR(wp2), TR(wp2 + 1024)); F0.w[3] = CAT(TR(wp3), TR(wp3 + 1024));
#pragma unroll
        for (int m = 0; m < 8; ++m) F0.x[m] = XR(m);
        ap = xa1; PIN(ap);
        __builtin_amdgcn_s_waitcnt(0xC07F);
        SB();
        for (int t = 0; t < NKS - 1; ++t) {
            BODY0();
            BODY1();
        }
        HEAD(F0, F1, 1);
#pragma unroll
        for (int i = 16; i < 32; ++i) MF(F0, i);
#pragma unroll
        for (int i = 0; i < 32; ++i) MF(F1, i);
        __builtin_amdgcn_s_setprio(0);
        END_STAMP();
        const bool bad = rfl(__hip_atomic_load(tmo, __ATOMIC_RELAXED, __HIP_MEMORY_SCOPE_AGENT)) == call;
        char* obase = P.out + (size_t)(mh * BM + wr * 128 + fr) * (NT * 2) + (size_t)((wc ? ch1 : ch0) * 64 + (g & 1) * 16 + (g >> 1) * 8) * 2;
        const __amdgpu_buffer_rsrc_t ors = __builtin_amdgcn_make_buffer_rsrc((void*)(P.out + (size_t)(mh * BM + wr * 128) * (NT * 2) + (size_t)((wc ? ch1 : ch0) * 64) * 2), 0, 128 * NT * 2 - (wc ? ch1 : ch0) * 128, 0x00020000);
        const unsigned ovo = fr * (NT * 2) + ((g & 1) * 16 + (g >> 1) * 8) * 2;
        if (__builtin_expect(bad, 0)) {
#pragma unroll 1
            for (int i = 0; i < 16; ++i)
                *(u32x4*)(obase + (size_t)(i >> 1) * 16 * (NT * 2) + (i & 1) * 64) = (u32x4){0x7fc07fc0u, 0x7fc07fc0u, 0x7fc07fc0u, 0x7fc07fc0u};
        } else {
#pragma unroll
            for (int m = 0; m < 8; ++m) {
#pragma unroll
                for (int n = 0; n < 4; n += 2) {
                    f32x4 a = acc[m][n], c = acc[m][n + 1];
                    unsigned a0 = gelu2(a[0], a[1]), a1 = gelu2(a[2], a[3]);
                    unsigned c0 = gelu2(c[0], c[1]), c1 = gelu2(c[2], c[3]);
                    auto r0 = __builtin_amdgcn_permlane16_swap(a0, c0, false, false);
                    auto r1 = __builtin_amdgcn_permlane16_swap(a1, c1, false, false);
                    u32x4 o = {r0[0], r1[0], r0[1], r1[1]};
                    __builtin_amdgcn_raw_buffer_store_b128(o, ors, ovo + m * 16 * (NT * 2) + n * 32, 0, CFG_OAUX);
                }
            }
        }
    } else if (wave < 6) {
        const int h = wave - 4;
        __builtin_amdgcn_s_setprio(CFG_PL);
        const unsigned vo0 = (lane >> 3) * 1024 + (((lane & 7) ^ (lane >> 4)) << 4);
        const unsigned vo1 = vo0 ^ 64u;
        const char* own = P.x + (size_t)rank * BLK_BYTES + (size_t)(mh * BM + h * 128) * 1024;
        LAS char* ring = L + A_OFF + h * NA * SLOT;
        int slot = 0;
        unsigned vo[16];
#pragma unroll
        for (int i = 0; i < 16; ++i) vo[i] = ((i & 1) ? vo1 : vo0) + i * 8192;
        auto issue_part = [&](const char* src, int i0, int i1) {
            __amdgpu_buffer_rsrc_t rs = __builtin_amdgcn_make_buffer_rsrc((void*)src, 0, A_REC, 0x00020000);
            LAS char* dst = ring + slot * SLOT;
#pragma unroll
            for (int i = 0; i < 16; ++i) if (i >= i0 && i < i1)
                DMA16(rs, (LAS void*)(dst + i * 1024), vo[i], 0, CFG_AUXA);
            if (i1 == 16) slot = (slot + 1 == NA) ? 0 : slot + 1;
        };
        auto issue = [&](const char* src) { issue_part(src, 0, 16); };
        unsigned cnt_old;
#pragma unroll
        for (int t = 0; t < CFG_PRO; ++t) issue(own + ((t + rot) & 7) * 128);
        WAIT_V(16 * (CFG_PRO - 1));
        BAR();
#pragma unroll
        for (int t = CFG_PRO; t < NA; ++t) issue(own + ((t + rot) & 7) * 128);
        asm volatile("global_load_dword %0, %1, off sc1" : "=v"(cnt_old) : "v"(cnt) : "memory");
        WAIT_V(17); BAR(); issue(own + ((3 + rot) & 7) * 128);
        asm volatile("s_waitcnt vmcnt(16)" : "+v"(cnt_old) :: "memory");
        const unsigned call = rfl(cnt_old) + 1u;
        const char* rem = myws + WS_DATA + (size_t)((call & 1u) * NR) * SRC_BYTES;
        Sched S; S.init(call, (P.mode & 2) != 0);
        const bool poller = (h == 0);
        const __amdgpu_buffer_rsrc_t frs = __builtin_amdgcn_make_buffer_rsrc((void*)(myws + WS_FLAG + (size_t)(b % CFG_R) * 1024), 0, 1024, 0x00020000);
        const unsigned fvo = ((src_of(lane) << 3) | (lane >> 3)) * 16;
        const unsigned long long t0 = __builtin_amdgcn_s_memrealtime();
        const char* src = own + ((4 + rot) & 7) * 128;
        bool have = true;
        for (int t = 1; t < NKS - 1; ++t) {
            const int tn = t + NA;
            if (t + 2 < NKS) WAIT_V(16); else WAIT_V(0);
            const unsigned sb = lbase + SB_OFF + (t & 1) * 1024 + lane * 16;
            if (__builtin_expect(poller && !have, 0)) {
                Sched T = S;
                T.update(lds_rd128(sb));
                if (T.total() < (unsigned)(tn - 7)) {
                    for (;;) {
                        __builtin_amdgcn_raw_ptr_buffer_load_lds(frs, (LAS void*)(L + SB_OFF + (t & 1) * 1024), 16, fvo, 0, 0, 17);
                        WAIT_V(0);
                        T.update(lds_rd128(sb));
                        if (T.total() >= (unsigned)(tn - 7)) break;
                        if (rfl(__hip_atomic_load(tmo, __ATOMIC_RELAXED, __HIP_MEMORY_SCOPE_AGENT)) == call ||
                            __builtin_amdgcn_s_memrealtime() - t0 > DEADLINE) {
                            if (lane == 0) __hip_atomic_store(tmo, call, __ATOMIC_RELAXED, __HIP_MEMORY_SCOPE_AGENT);
                            const u32x4 full = {S.need, S.need, S.need, S.need};
                            asm volatile("ds_write_b128 %0, %1\n\ts_waitcnt vmcnt(0) lgkmcnt(0)" :: "v"(sb), "v"(full) : "memory");
                            break;
                        }
                        __builtin_amdgcn_s_sleep(8);
                    }
                }
            }
            BAR();
            if (tn < NKS) {
                if (tn == 8) {
#pragma unroll
                    for (int i = 0; i < 16; ++i)
                        vo[i] = stage_off(mh * BM + h * 128 + 8 * i + (lane >> 3)) + ((((lane & 7) ^ (lane >> 4)) ^ ((i & 1) << 2)) << 4);
                }
                if (__builtin_expect(!have, 0)) {
                    int u;
                    S.update(lds_rd128(sb));
                    S.next(u);
                    src = rem + (size_t)src_of(u) * SRC_BYTES + (u >> 3) * UNIT_BYTES;
                }
                if (poller && tn + 2 >= 8 && tn + 1 < NKS && !S.all() && S.spare() <= (unsigned)CFG_LOW)
                    __builtin_amdgcn_raw_ptr_buffer_load_lds(frs, (LAS void*)(L + SB_OFF + ((t + 1) & 1) * 1024), 16, fvo, 0, 0, 17);
                issue_part(src, 0, 6);
                const char* nsrc = own + ((tn + 1 + rot) & 7) * 128;
                have = true;
                if (tn + 1 >= 8 && tn + 1 < NKS) {
                    int u;
                    if (!S.all()) S.update(lds_rd128(sb));
                    have = S.next(u);
                    nsrc = rem + (size_t)src_of(u) * SRC_BYTES + (u >> 3) * UNIT_BYTES;
                }
                issue_part(src, 6, 16);
                src = nsrc;
            }
        }
        END_STAMP();
    } else if (wave == 6) {
        const int l4 = lane >> 4;
        __builtin_amdgcn_s_setprio(CFG_PL);
        const unsigned c0 = (lane & 15) ^ (l4 << 1), c1 = c0 ^ 8u;
        const unsigned cd = (unsigned)(ch1 - ch0) * 128u;
        const unsigned vo0 = l4 * (NT * 2) + (c0 & 7u) * 16u + (c0 >> 3) * cd;
        const unsigned vo1 = l4 * (NT * 2) + (c1 & 7u) * 16u + (c1 >> 3) * cd;
        const char* wb = P.w + (size_t)ch0 * 128;
        LAS char* ring = L + W_OFF;
        int slot = 0;
        auto issue_part = [&](int krow, int i0, int i1) {
            __amdgpu_buffer_rsrc_t rs = __builtin_amdgcn_make_buffer_rsrc((void*)(wb + (size_t)krow * (NT * 2)), 0, W_REC, 0x00020000);
            LAS char* dst = ring + slot * SLOT;
#pragma unroll
            for (int i = 0; i < 16; ++i) if (i >= i0 && i < i1)
                DMA16(rs, (LAS void*)(dst + i * 1024), (i & 2) ? vo1 : vo0, i * 4 * NT * 2, CFG_AUXW);
            if (i1 == 16) slot = (slot + 1 == NW) ? 0 : slot + 1;
        };
        auto issue = [&](int krow) { issue_part(krow, 0, 16); };
        unsigned cnt_old;
#pragma unroll
        for (int t = 0; t < CFG_PRO; ++t) issue(rank * KB + ((t + rot) & 7) * 64);
        WAIT_V(16 * (CFG_PRO - 1));
        BAR();
#pragma unroll
        for (int t = CFG_PRO; t < NW; ++t) issue(rank * KB + ((t + rot) & 7) * 64);
        asm volatile("global_load_dword %0, %1, off sc1" : "=v"(cnt_old) : "v"(cnt) : "memory");
        WAIT_V(16 * (NW - 2) + 1); BAR(); issue(rank * KB + ((NW + rot) & 7) * 64);
        asm volatile("s_waitcnt vmcnt(16)" : "+v"(cnt_old) :: "memory");
        const unsigned call = rfl(cnt_old) + 1u;
        Sched S; S.init(call, (P.mode & 2) != 0);
        int krow = rank * KB + ((NW + 1 + rot) & 7) * 64;
        bool have = true;
        for (int t = 1; t < NKS - 1; ++t) {
            const int tn = t + NW;
            if (NW == 4 && t + 3 < NKS) WAIT_V(32); else if (t + 2 < NKS) WAIT_V(16); else WAIT_V(0);
            const unsigned sb = lbase + SB_OFF + (t & 1) * 1024 + lane * 16;
            BAR();
            if (tn < NKS) {
                if (__builtin_expect(!have, 0)) {
                    int u;
                    S.update(lds_rd128(sb));
                    S.next(u);
                    krow = src_of(u) * KB + (u >> 3) * 64;
                }
                issue_part(krow, 0, 6);
                int nk = rank * KB + ((tn + 1 + rot) & 7) * 64;
                have = true;
                if (tn + 1 >= 8 && tn + 1 < NKS) {
                    int u;
                    if (!S.all()) S.update(lds_rd128(sb));
                    have = S.next(u);
                    nk = src_of(u) * KB + (u >> 3) * 64;
                }
                issue_part(krow, 6, 16);
                krow = nk;
            }
        }
        END_STAMP();
    } else {
        const bool pusher = (b < 224) && !(P.mode & 1);
        const int dest = (rank + 1 + (b >> 5)) & 7;
        const int quarter = b & 3, stage = (((b & 31) >> 2) + (b >> 5)) & 7;
        u32x4 v[16];
        const unsigned voff = (lane >> 3) * 1024 + (lane & 7) * 16;
        const unsigned long long t0 = __builtin_amdgcn_s_memrealtime();
        const unsigned cnt_old = __hip_atomic_load(cnt, __ATOMIC_RELAXED, __HIP_MEMORY_SCOPE_AGENT);
        if (pusher) {
            const char* src = P.x + (size_t)dest * BLK_BYTES + (size_t)quarter * 131072 + stage * 128 + voff;
#pragma unroll
            for (int i = 0; i < 16; ++i) v[i] = *(const u32x4*)(src + i * 8192);
        }
        asm volatile("buffer_inv sc0 sc1" ::: "memory");
#define CFG_EARLY 1
        int t = -1;
        if (!(pusher && stage < CFG_EARLY))
            while (t < 2 && (t < 0 || vm_outstanding() != 0u)) { BAR(); ++t; }
        WAIT_V(0);
        const unsigned call = rfl(cnt_old) + 1u;
        char* dws = P.ws[dest];
        __amdgpu_buffer_rsrc_t rs = __builtin_amdgcn_make_buffer_rsrc(
            (void*)(dws + WS_DATA + (size_t)((call & 1u) * NR + rank) * SRC_BYTES + (size_t)stage * UNIT_BYTES), 0, UNIT_BYTES, 0x00020000);
        unsigned* fl = (unsigned*)(dws + WS_FLAG) + (rank * 8 + stage) * 4 + quarter;
        const char* myflag = myws + WS_FLAG + (size_t)(b % CFG_R) * 1024 + ((src_of(lane) << 3) | (lane >> 3)) * 16;
        Sched V; V.init(call, (P.mode & 2) != 0);
        const unsigned long long tgo = t0 + (unsigned long long)(CFG_TB + stage * CFG_TS);
#define CFG_Q 16
#define CFG_PD 0
        int nst = 0;
        bool go = (stage < CFG_EARLY), pending = pusher;
        const __amdgpu_buffer_rsrc_t wrs = __builtin_amdgcn_make_buffer_rsrc((void*)(P.w + (size_t)np * BN * 2), 0, (unsigned)((size_t)KT * NT * 2 - (size_t)np * BN * 2), 0x00020000);
        const __amdgpu_buffer_rsrc_t ars = __builtin_amdgcn_make_buffer_rsrc((void*)(myws + WS_DATA + (size_t)((call & 1u) * NR) * SRC_BYTES), 0, (unsigned)(NR * SRC_BYTES), 0x00020000);
        const unsigned wvo = (unsigned)(mh * 32 + (lane >> 1)) * (NT * 2) + (lane & 1) * 128;
        const unsigned avo = stage_off(bi * 16 + (lane & 15));
        unsigned long long wtk = OWN_BITS, atk = OWN_BITS;
        int wcnt = NA + 1;
        for (;;) {
            bool busy = false;
            if (pending) {
                if (!go) go = __builtin_amdgcn_s_memrealtime() >= tgo;
                if (go) {
                    busy = true;
                    unsigned out_now = vm_outstanding();
                    if (nst == 16 && out_now == 0u) {
                        WAIT_V(0);
                        if (lane < CFG_R) __hip_atomic_store(fl + lane * 256, call, __ATOMIC_RELAXED, __HIP_MEMORY_SCOPE_SYSTEM);
                        pending = false; busy = false;
                    }
                    while (nst < 16 && out_now + 4u <= (unsigned)CFG_Q + 24u) {
#define PST(i) case i: _Pragma("unroll") for (int k = 0; k < 4; ++k) __builtin_amdgcn_raw_buffer_store_b128(v[(i) + k], rs, stage_off(quarter * 128 + 8 * ((i) + k) + (lane >> 3)) + (lane & 7) * 16, 0, 17); break;
                        switch (nst) { PST(0) PST(4) PST(8) default: PST(12) }
#undef PST
                        out_now += 4u; nst += 4;
                    }
                }
            }
            if (t >= NKS - 1) { if (!pending) break; __builtin_amdgcn_s_sleep(1); continue; }
            const int tn = t + NA;
            if (tn >= 8 && !V.all()) V.update(lds_rd128(lbase + SB_OFF + ((t - 1) & 1) * 1024 + lane * 16));
            if (pending && tn >= 8 && tn < NKS && V.total() < (unsigned)(tn - 7)) {
                u32x4 f;
                asm volatile("global_load_dwordx4 %0, %1, off sc0 sc1\n\ts_waitcnt vmcnt(0)" : "=v"(f) : "v"(myflag) : "memory");
                V.update(f);
                if (V.total() < (unsigned)(tn - 7)) {
                    if (rfl(__hip_atomic_load(tmo, __ATOMIC_RELAXED, __HIP_MEMORY_SCOPE_AGENT)) == call ||
                        __builtin_amdgcn_s_memrealtime() - t0 > DEADLINE) V.avail = ~0ull;
                    WAIT_V(0);
                    __builtin_amdgcn_s_sleep(4);
                    continue;
                }
            }
            BAR(); ++t;
            if (t == 3 && lane == 0) __hip_atomic_store(cnt, call, __ATOMIC_RELAXED, __HIP_MEMORY_SCOPE_AGENT);
            if (!pending && t > 3 && (CFG_PD == 0 || (wcnt >= NKS && (V.avail & ~atk) == 0ull && V.all()))) break;
        }
        WAIT_V(0);
    }
}

extern "C" void kernel_launch(int rank, int nranks, ncclComm_t comm,
                              hipStream_t stream,
                              void* const* d_in, const int* in_sizes, int n_in,
                              void* d_out, int out_size,
                              const int* in_dtypes, int out_dtype,
                              void* const* d_out_all,
                              void* const* d_ws_all, size_t ws_bytes) {
    using namespace cfg;
    (void)comm; (void)in_dtypes; (void)out_dtype; (void)d_out_all;
    if (nranks != NR || n_in != 2 || (unsigned)rank >= (unsigned)NR || ws_bytes < WS_NEED ||
        in_sizes[0] != KT * KB || in_sizes[1] != KT * NT || out_size != ML * NT) {
        fprintf(stderr, "kernel_launch: unexpected problem shape\n");
        return;
    }
    static bool attr_set = false;
    if (!attr_set) {
        HIP_CHECK(hipFuncSetAttribute((const void*)a2a_gemm_kernel, hipFuncAttributeMaxDynamicSharedMemorySize, LDS_BYTES));
        attr_set = true;
    }
    Params P;
    P.x = (const char*)d_in[0];
    P.w = (const char*)d_in[1];
    P.out = (char*)d_out;
    for (int p = 0; p < NR; ++p) P.ws[p] = (char*)d_ws_all[p];
    P.rank = rank;
    P.mode = 0;
    P.dbg = nullptr;
    hipLaunchKernelGGL(a2a_gemm_kernel, dim3(NWG), dim3(512), LDS_BYTES, stream, P);
    HIP_CHECK(hipGetLastError());
}
```

```cpp
#include <hip/hip_runtime.h>
#include <rccl/rccl.h>
#include <cstdio>
#include <cstdint>

#define HIP_CHECK(expr) do { hipError_t _e = (expr); if (_e != hipSuccess) { \
    fprintf(stderr, "HIP error %s at %s:%d\n", hipGetErrorString(_e), __FILE__, __LINE__); \
    return; } } while (0)

#define LAS __attribute__((address_space(3)))
#define DI __device__ __forceinline__
typedef __bf16 bf16x8 __attribute__((ext_vector_type(8)));
typedef __bf16 bf16x4 __attribute__((ext_vector_type(4)));
typedef __bf16 bf16x2 __attribute__((ext_vector_type(2)));
typedef float f32x4 __attribute__((ext_vector_type(4)));
typedef float f32x2 __attribute__((ext_vector_type(2)));
typedef unsigned u32x4 __attribute__((ext_vector_type(4)));

namespace cfg {
constexpr int NR = 8;
constexpr int ML = 512;
constexpr int KB = 512;
constexpr int KT = 4096, NT = 16384;
constexpr int BM = 256, BK = 64;
constexpr int NKS = KT / BK;
constexpr int NWG = 256;
constexpr int NS = 3;
constexpr int SLOT = 16384;
constexpr int A_OFF = 0;
constexpr int W_OFF = 2 * NS * SLOT;
constexpr int SB_OFF = W_OFF + NS * SLOT;
constexpr int LDS_BYTES = SB_OFF + 2048;
static_assert(LDS_BYTES <= 163840, "LDS");
constexpr int NREP = 16;
constexpr size_t WS_CNT = 0;
constexpr size_t WS_TMO = 4096;
constexpr size_t WS_FLAG = 8192;
constexpr size_t WS_DATA = 1u << 20;
constexpr size_t BLK_BYTES = (size_t)ML * KB * 2;
constexpr unsigned UNIT_BYTES = 74 * 1024;
constexpr size_t SRC_BYTES = (size_t)8 * UNIT_BYTES;
constexpr size_t WS_NEED = WS_DATA + 2 * NR * SRC_BYTES;
static_assert(WS_FLAG + NREP * 1024 <= WS_DATA, "flags");
#define CFG_DEADLINE 150000000ull
constexpr unsigned long long DEADLINE = CFG_DEADLINE;
#define CFG_TS 150
#define CFG_TB 250
constexpr int PUSH_T0 = CFG_TB, PUSH_TS = CFG_TS;
}

__host__ __device__ inline unsigned stage_off(unsigned r) {
    const unsigned g = r / 7u, s = r - 7u * g;
    return g * 1024u + (s + (s >= 3u ? 1u : 0u)) * 128u;
}

struct Params {
    const char* x;
    const char* w;
    char* out;
    char* ws[cfg::NR];
    int rank;
    int mode;
    unsigned long long* dbg;
};

DI unsigned rfl(unsigned v) { return __builtin_amdgcn_readfirstlane(v); }
#define WAIT_V(n) asm volatile("s_waitcnt vmcnt(%0)" :: "n"(n) : "memory")
#define BAR() do { asm volatile("" ::: "memory"); __builtin_amdgcn_s_barrier(); asm volatile("" ::: "memory"); } while (0)
DI unsigned vm_outstanding() {
    const unsigned r = __builtin_amdgcn_s_getreg(7 | (0 << 6) | (31 << 11));
    return (r & 15u) | ((r >> 18) & 48u);
}
DI u32x4 lds_rd128(unsigned addr) { u32x4 v; asm volatile("ds_read_b128 %0, %1\n\ts_waitcnt lgkmcnt(0)" : "=v"(v) : "v"(addr) : "memory"); return v; }

DI unsigned gelu2(float y0, float y1) {
    const float k1 = -2.0f * 0.7978845608028654f * 1.4426950408889634f;
    const float k3 = k1 * 0.044715f;
    const f32x2 y = {y0, y1};
    const f32x2 z = y * __builtin_elementwise_fma(y * y, (f32x2){k3, k3}, (f32x2){k1, k1});
    f32x2 e; e.x = __builtin_amdgcn_exp2f(z.x); e.y = __builtin_amdgcn_exp2f(z.y);
    const f32x2 d = e + 1.0f;
    f32x2 r; r.x = __builtin_amdgcn_rcpf(d.x); r.y = __builtin_amdgcn_rcpf(d.y);
    const f32x2 o = y * r;
    const bf16x2 bb = __builtin_convertvector(o, bf16x2);
    return __builtin_bit_cast(unsigned, bb);
}

#define MFMA1(c, a, b) c = __builtin_amdgcn_mfma_f32_16x16x32_bf16(a, b, c, 0, 0, 0)
#define DMA16(rs, dst, vo, so, aux) __builtin_amdgcn_raw_ptr_buffer_load_lds(rs, dst, 16, vo, so, 0, aux)
#define MKRS(p, n) __builtin_amdgcn_make_buffer_rsrc((void*)(p), 0, (n), 0x00020000)
struct Frags { bf16x8 x[8]; bf16x8 w[4]; };

constexpr unsigned long long OWN_BITS = 0x8080808080808080ull;
struct Sched {
    unsigned long long avail, taken;
    unsigned need;
    DI void init(unsigned call, bool all) { need = call; taken = OWN_BITS; avail = all ? ~0ull : OWN_BITS; }
    DI bool all() const { return avail == ~0ull; }
    DI void update(u32x4 f) {
        unsigned m = f.x < f.y ? f.x : f.y, m2 = f.z < f.w ? f.z : f.w;
        m = m < m2 ? m : m2;
        avail |= __ballot(m >= need);
    }
    DI unsigned spare() const { return (unsigned)__builtin_popcountll(avail & ~taken); }
    DI unsigned total() const { return (unsigned)__builtin_popcountll(avail) - 8u; }
    DI bool next(int& idx) {
        const unsigned long long c = avail & ~taken;
        if (c == 0ull) { idx = 0; return false; }
        idx = __builtin_ctzll(c);
        taken |= 1ull << idx;
        return true;
    }
};

__global__ __launch_bounds__(512, 2) void a2a_gemm_kernel(Params P) {
    using namespace cfg;
    extern __shared__ __attribute__((aligned(16))) char shm[];
    LAS char* L = (LAS char*)shm;
    const unsigned lbase = (unsigned)(size_t)L;
    const int tid = threadIdx.x, lane = tid & 63;
    const int wave = __builtin_amdgcn_readfirstlane(tid >> 6);
    const int b = blockIdx.x;
    const int xcd = b & 7, bi = b >> 3;
    const int mh = bi & 1;
    const int np = xcd * 16 + (bi >> 1);
    const int ch0 = (np >> 2) * 8 + (np & 2) * 2 + (np & 1), ch1 = ch0 + 2;
    const int rank = P.rank;
    const int rot = (xcd + 3 * mh) & 7, rot7 = (xcd + 3 * mh) % 7;
    auto src_of = [&](int u) { int q = (u & 7) + rot7; q = q >= 7 ? q - 7 : q; return (u & 7) == 7 ? rank : ((rank + 1 + q) & 7); };
    char* const myws = P.ws[rank];
    unsigned* const cnt = (unsigned*)(myws + WS_CNT) + b;
    unsigned* const tmo = (unsigned*)(myws + WS_TMO);
    const unsigned fvo = (unsigned)((src_of(lane) << 3) | (lane >> 3)) * 16u;
    const char* const myflags = myws + WS_FLAG + (size_t)(b % NREP) * 1024;

    if (wave < 4) {
        const int wr = wave >> 1, wc = wave & 1;
        const unsigned call = rfl(__hip_atomic_load(cnt, __ATOMIC_RELAXED, __HIP_MEMORY_SCOPE_AGENT)) + 1u;
        const int fr = lane & 15, g = lane >> 4;
        const unsigned xa0 = lbase + A_OFF + wr * NS * SLOT + fr * 128 + ((g ^ (fr >> 1)) << 4);
        const unsigned xa1 = xa0 ^ 64u;
        const int q = (lane & 15) >> 2, pp = lane & 3, gl = q | ((g & 1) << 2);
        unsigned wa[4];
#pragma unroll
        for (int n = 0; n < 4; ++n) wa[n] = lbase + W_OFF + (8 * g + q) * 256 + (((wc * 4 + n) ^ gl) << 5) + pp * 8;

        f32x4 acc[8][4];
#pragma unroll
        for (int m = 0; m < 8; ++m)
#pragma unroll
            for (int n = 0; n < 4; ++n) acc[m][n] = (f32x4){0.f, 0.f, 0.f, 0.f};

        Frags F0, F1;
        unsigned slot = 0;
        unsigned ap, wp0, wp1, wp2, wp3;
#define PIN(x) asm volatile("" : "+v"(x))
#define SB() __builtin_amdgcn_sched_barrier(0)
#define LP(a) ((LAS char*)(size_t)(a))
#define TR(a) __builtin_amdgcn_ds_read_tr16_b64_v4bf16((LAS bf16x4*)LP(a))
#define CAT(lo, hi) __builtin_shufflevector(lo, hi, 0, 1, 2, 3, 4, 5, 6, 7)
#define XR(m) (*(LAS bf16x8*)LP(ap + (m) * 2048))
#define NI(i) ((((i) >> 2) & 1) ? 3 - ((i) & 3) : ((i) & 3))
#define MF(FC, i) MFMA1(acc[(i) >> 2][NI(i)], FC.w[NI(i)], FC.x[(i) >> 2])
#define HEAD(FC, FN, KH) do { \
            bf16x4 lo_, hi_; \
            MF(FC, 0);  lo_ = TR(wp0 + (KH) * 8192);        SB(); \
            MF(FC, 1);  hi_ = TR(wp0 + (KH) * 8192 + 1024); FN.w[0] = CAT(lo_, hi_); SB(); \
            MF(FC, 2);  FN.x[0] = XR(0); SB(); \
            MF(FC, 3);  lo_ = TR(wp1 + (KH) * 8192);        SB(); \
            MF(FC, 4);  hi_ = TR(wp1 + (KH) * 8192 + 1024); FN.w[1] = CAT(lo_, hi_); SB(); \
            MF(FC, 5);  lo_ = TR(wp2 + (KH) * 8192);        SB(); \
            MF(FC, 6);  hi_ = TR(wp2 + (KH) * 8192 + 1024); FN.w[2] = CAT(lo_, hi_); SB(); \
            MF(FC, 7);  lo_ = TR(wp3 + (KH) * 8192);        SB(); \
            MF(FC, 8);  hi_ = TR(wp3 + (KH) * 8192 + 1024); FN.w[3] = CAT(lo_, hi_); SB(); \
            MF(FC, 9);  FN.x[1] = XR(1); SB(); \
            MF(FC, 10); FN.x[2] = XR(2); SB(); \
            MF(FC, 11); FN.x[3] = XR(3); SB(); \
            MF(FC, 12); FN.x[4] = XR(4); SB(); \
            MF(FC, 13); FN.x[5] = XR(5); SB(); \
            MF(FC, 14); FN.x[6] = XR(6); SB(); \
            MF(FC, 15); FN.x[7] = XR(7); SB(); \
            } while (0)
#define BODY0() do { \
            HEAD(F0, F1, 1); \
            MF(F0, 16); slot += SLOT; if (slot == NS * SLOT) slot = 0; SB(); \
            MF(F0, 17); ap = xa0 + slot; PIN(ap); wp0 = wa[0] + slot; PIN(wp0); SB(); \
            MF(F0, 18); wp1 = wa[1] + slot; PIN(wp1); wp2 = wa[2] + slot; PIN(wp2); SB(); \
            MF(F0, 19); wp3 = wa[3] + slot; PIN(wp3); SB(); \
            _Pragma("unroll") for (int i_ = 20; i_ < 32; ++i_) { MF(F0, i_); SB(); \
                if (i_ == 27) { __builtin_amdgcn_s_waitcnt(0xC07F); BAR(); SB(); } } \
            } while (0)
#define BODY1() do { \
            HEAD(F1, F0, 0); \
            MF(F1, 16); ap = xa1 + slot; PIN(ap); SB(); \
            _Pragma("unroll") for (int i_ = 17; i_ < 32; ++i_) { MF(F1, i_); SB(); if (i_ == 29) { __builtin_amdgcn_s_waitcnt(0xC07F); SB(); } } \
            } while (0)

        ap = xa0; wp0 = wa[0]; wp1 = wa[1]; wp2 = wa[2]; wp3 = wa[3];
        BAR();
        F0.w[0] = CAT(TR(wp0), TR(wp0 + 1024)); F0.w[1] = CAT(TR(wp1), TR(wp1 + 1024));
        F0.w[2] = CAT(TR(wp2), TR(wp2 + 1024)); F0.w[3] = CAT(TR(wp3), TR(wp3 + 1024));
#pragma unroll
        for (int m = 0; m < 8; ++m) F0.x[m] = XR(m);
        ap = xa1; PIN(ap);
        __builtin_amdgcn_s_waitcnt(0xC07F);
        SB();
        for (int t = 0; t < NKS - 1; ++t) {
            BODY0();
            BODY1();
        }
        const unsigned tmo_v = __hip_atomic_load(tmo, __ATOMIC_RELAXED, __HIP_MEMORY_SCOPE_AGENT);
        const int chw = wc ? ch1 : ch0;
        const __amdgpu_buffer_rsrc_t ors = MKRS(P.out + (size_t)(mh * BM + wr * 128) * (NT * 2) + (size_t)chw * 128, 128 * NT * 2 - chw * 128);
        const unsigned ovo = fr * (NT * 2) + ((g & 1) * 16 + (g >> 1) * 8) * 2;
        HEAD(F0, F1, 1);
        const bool bad = rfl(tmo_v) == call;
        if (__builtin_expect(bad, 0)) {
            const u32x4 qn = {0x7fc07fc0u, 0x7fc07fc0u, 0x7fc07fc0u, 0x7fc07fc0u};
#pragma unroll 1
            for (int i = 0; i < 16; ++i)
                __builtin_amdgcn_raw_buffer_store_b128(qn, ors, ovo + (i >> 1) * 16 * (NT * 2) + (i & 1) * 64, 0, 0);
        } else {
#define FIN(m) do { _Pragma("unroll") for (int n_ = 0; n_ < 4; ++n_) { \
                if ((m) >= 4) MFMA1(acc[m][n_], F0.w[n_], F0.x[m]); \
                MFMA1(acc[m][n_], F1.w[n_], F1.x[m]); } } while (0)
#define EPI(m) do { _Pragma("unroll") for (int n = 0; n < 4; n += 2) { \
                const f32x4 a = acc[m][n], c = acc[m][n + 1]; \
                unsigned a0 = gelu2(a[0], a[1]), a1 = gelu2(a[2], a[3]); \
                unsigned c0 = gelu2(c[0], c[1]), c1 = gelu2(c[2], c[3]); \
                auto r0 = __builtin_amdgcn_permlane16_swap(a0, c0, false, false); \
                auto r1 = __builtin_amdgcn_permlane16_swap(a1, c1, false, false); \
                u32x4 o = {r0[0], r1[0], r0[1], r1[1]}; \
                __builtin_amdgcn_raw_buffer_store_b128(o, ors, ovo + (m) * 16 * (NT * 2) + n * 32, 0, 0); } } while (0)
            FIN(0); FIN(1); SB();
            FIN(2); EPI(0); SB();
            FIN(3); EPI(1); SB();
            FIN(4); EPI(2); SB();
            FIN(5); EPI(3); SB();
            FIN(6); EPI(4); SB();
            FIN(7); EPI(5); SB();
            EPI(6); EPI(7);
        }
    } else if (wave < 6) {
        const int h = wave - 4;
        const char* own = P.x + (size_t)rank * BLK_BYTES + (size_t)(mh * BM + h * 128) * 1024;
        LAS char* ring = L + A_OFF + h * NS * SLOT;
        int slot = 0;
        unsigned vo[16];
#pragma unroll
        for (int i = 0; i < 16; ++i) vo[i] = (8 * i + (lane >> 3)) * 1024 + ((((lane & 7) ^ (lane >> 4)) ^ ((i & 1) << 2)) << 4);
        auto issue_part = [&](const char* src, int i0, int i1) {
            const __amdgpu_buffer_rsrc_t rs = MKRS(src, 131072);
            LAS char* dst = ring + slot * SLOT;
#pragma unroll
            for (int i = 0; i < 16; ++i) if (i >= i0 && i < i1)
                DMA16(rs, (LAS void*)(dst + i * 1024), vo[i], 0, 0);
            if (i1 == 16) slot = (slot + 1 == NS) ? 0 : slot + 1;
        };
        auto issue = [&](const char* src) { issue_part(src, 0, 16); };
        unsigned cnt_old;
        issue(own + (rot & 7) * 128); issue(own + ((1 + rot) & 7) * 128);
        WAIT_V(16);
        BAR();
        issue(own + ((2 + rot) & 7) * 128);
        asm volatile("global_load_dword %0, %1, off sc1" : "=v"(cnt_old) : "v"(cnt) : "memory");
        WAIT_V(17); BAR(); issue(own + ((3 + rot) & 7) * 128);
        asm volatile("s_waitcnt vmcnt(16)" : "+v"(cnt_old) :: "memory");
        const unsigned call = rfl(cnt_old) + 1u;
        const char* rem = myws + WS_DATA + (size_t)((call & 1u) * NR) * SRC_BYTES;
        Sched S; S.init(call, (P.mode & 2) != 0);
        const bool poller = (h == 0);
        const __amdgpu_buffer_rsrc_t frs = MKRS(myflags, 1024);
        const unsigned long long t0 = __builtin_amdgcn_s_memrealtime();
        const char* src = own + ((4 + rot) & 7) * 128;
        bool have = true;
        for (int t = 1; t < NKS - 1; ++t) {
            const int tn = t + NS;
            if (t + 2 < NKS) WAIT_V(16); else WAIT_V(0);
            const unsigned sb = lbase + SB_OFF + (t & 1) * 1024 + lane * 16;
            if (__builtin_expect(poller && !have, 0)) {
                Sched T = S;
                T.update(lds_rd128(sb));
                while (T.total() < (unsigned)(tn - 7)) {
                    if (rfl(__hip_atomic_load(tmo, __ATOMIC_RELAXED, __HIP_MEMORY_SCOPE_AGENT)) == call ||
                        __builtin_amdgcn_s_memrealtime() - t0 > DEADLINE) {
                        if (lane == 0) __hip_atomic_store(tmo, call, __ATOMIC_RELAXED, __HIP_MEMORY_SCOPE_AGENT);
                        const u32x4 full = {S.need, S.need, S.need, S.need};
                        asm volatile("ds_write_b128 %0, %1\n\ts_waitcnt vmcnt(0) lgkmcnt(0)" :: "v"(sb), "v"(full) : "memory");
                        break;
                    }
                    DMA16(frs, (LAS void*)(L + SB_OFF + (t & 1) * 1024), fvo, 0, 17);
                    WAIT_V(0);
                    T.update(lds_rd128(sb));
                    if (T.total() < (unsigned)(tn - 7)) __builtin_amdgcn_s_sleep(8);
                }
            }
            BAR();
            if (tn < NKS) {
                if (tn == 8) {
#pragma unroll
                    for (int i = 0; i < 16; ++i)
                        vo[i] = stage_off(mh * BM + h * 128 + 8 * i + (lane >> 3)) + ((((lane & 7) ^ (lane >> 4)) ^ ((i & 1) << 2)) << 4);
                }
                if (__builtin_expect(!have, 0)) {
                    int u;
                    S.update(lds_rd128(sb));
                    S.next(u);
                    src = rem + (size_t)src_of(u) * SRC_BYTES + (u >> 3) * UNIT_BYTES;
                }
                if (poller && tn + 2 >= 8 && tn + 1 < NKS && !S.all() && S.spare() <= 2u)
                    DMA16(frs, (LAS void*)(L + SB_OFF + ((t + 1) & 1) * 1024), fvo, 0, 17);
                issue_part(src, 0, 6);
                const char* nsrc = own + ((tn + 1 + rot) & 7) * 128;
                have = true;
                if (tn + 1 >= 8 && tn + 1 < NKS) {
                    int u;
                    if (!S.all()) S.update(lds_rd128(sb));
                    have = S.next(u);
                    nsrc = rem + (size_t)src_of(u) * SRC_BYTES + (u >> 3) * UNIT_BYTES;
                }
                issue_part(src, 6, 16);
                src = nsrc;
            }
        }
    } else if (wave == 6) {
        const int l4 = lane >> 4;
        const unsigned c0 = (lane & 15) ^ (l4 << 1), c1 = c0 ^ 8u;
        const unsigned cd = (unsigned)(ch1 - ch0) * 128u;
        const unsigned vo0 = l4 * (NT * 2) + (c0 & 7u) * 16u + (c0 >> 3) * cd;
        const unsigned vo1 = l4 * (NT * 2) + (c1 & 7u) * 16u + (c1 >> 3) * cd;
        const char* wb = P.w + (size_t)ch0 * 128;
        LAS char* ring = L + W_OFF;
        int slot = 0;
        auto issue_part = [&](int krow, int i0, int i1) {
            const __amdgpu_buffer_rsrc_t rs = MKRS(wb + (size_t)krow * (NT * 2), 63 * NT * 2 + 384);
            LAS char* dst = ring + slot * SLOT;
#pragma unroll
            for (int i = 0; i < 16; ++i) if (i >= i0 && i < i1)
                DMA16(rs, (LAS void*)(dst + i * 1024), (i & 2) ? vo1 : vo0, i * 4 * NT * 2, 0);
            if (i1 == 16) slot = (slot + 1 == NS) ? 0 : slot + 1;
        };
        auto issue = [&](int krow) { issue_part(krow, 0, 16); };
        unsigned cnt_old;
        issue(rank * KB + (rot & 7) * 64); issue(rank * KB + ((1 + rot) & 7) * 64);
        WAIT_V(16);
        BAR();
        issue(rank * KB + ((2 + rot) & 7) * 64);
        asm volatile("global_load_dword %0, %1, off sc1" : "=v"(cnt_old) : "v"(cnt) : "memory");
        WAIT_V(17); BAR(); issue(rank * KB + ((3 + rot) & 7) * 64);
        asm volatile("s_waitcnt vmcnt(16)" : "+v"(cnt_old) :: "memory");
        const unsigned call = rfl(cnt_old) + 1u;
        Sched S; S.init(call, (P.mode & 2) != 0);
        int krow = rank * KB + ((4 + rot) & 7) * 64;
        bool have = true;
        for (int t = 1; t < NKS - 1; ++t) {
            const int tn = t + NS;
            if (t + 2 < NKS) WAIT_V(16); else WAIT_V(0);
            const unsigned sb = lbase + SB_OFF + (t & 1) * 1024 + lane * 16;
            BAR();
            if (tn < NKS) {
                if (__builtin_expect(!have, 0)) {
                    int u;
                    S.update(lds_rd128(sb));
                    S.next(u);
                    krow = src_of(u) * KB + (u >> 3) * 64;
                }
                issue_part(krow, 0, 6);
                int nk = rank * KB + ((tn + 1 + rot) & 7) * 64;
                have = true;
                if (tn + 1 >= 8 && tn + 1 < NKS) {
                    int u;
                    if (!S.all()) S.update(lds_rd128(sb));
                    have = S.next(u);
                    nk = src_of(u) * KB + (u >> 3) * 64;
                }
                issue_part(krow, 6, 16);
                krow = nk;
            }
        }
    } else {
        const bool pusher = (b < 224) && !(P.mode & 1);
        const int dest = (rank + 1 + (b >> 5)) & 7;
        const int quarter = b & 3, stage = (((b & 31) >> 2) + (b >> 5)) & 7;
        const unsigned long long t0 = __builtin_amdgcn_s_memrealtime();
        const unsigned long long tgo = t0 + (unsigned long long)(stage == 0 ? 0 : PUSH_T0 + stage * PUSH_TS);
        const char* xsrc = P.x + (size_t)dest * BLK_BYTES + (size_t)quarter * 131072 + stage * 128 + (lane >> 3) * 1024 + (lane & 7) * 16;
        u32x4 v[16];
        const unsigned cnt_old = __hip_atomic_load(cnt, __ATOMIC_RELAXED, __HIP_MEMORY_SCOPE_AGENT);
        if (pusher) {
#pragma unroll
            for (int i = 0; i < 16; ++i) v[i] = *(const u32x4*)(xsrc + i * 8192);
        }
        asm volatile("buffer_inv sc0 sc1" ::: "memory");
        int t = -1;
        if (!(pusher && stage == 0))
            while (t < 2 && (t < 0 || vm_outstanding() != 0u)) { BAR(); ++t; }
        WAIT_V(0);
        const unsigned call = rfl(cnt_old) + 1u;
        if (pusher) {
            Sched V; V.init(call, (P.mode & 2) != 0);
            auto attend = [&]() {
                if (t >= NKS - 1) { __builtin_amdgcn_s_sleep(1); return; }
                const int tn = t + NS;
                if (tn >= 8 && tn < NKS && V.total() < (unsigned)(tn - 7)) {
                    V.update(lds_rd128(lbase + SB_OFF + ((t - 1) & 1) * 1024 + lane * 16));
                    if (V.total() < (unsigned)(tn - 7)) {
                        u32x4 f;
                        asm volatile("global_load_dwordx4 %0, %1, off sc0 sc1\n\ts_waitcnt vmcnt(0)" : "=v"(f) : "v"(myflags + fvo) : "memory");
                        V.update(f);
                        if (V.total() < (unsigned)(tn - 7)) {
                            if (rfl(__hip_atomic_load(tmo, __ATOMIC_RELAXED, __HIP_MEMORY_SCOPE_AGENT)) == call ||
                                __builtin_amdgcn_s_memrealtime() - t0 > DEADLINE) V.avail = ~0ull;
                            WAIT_V(0);
                            __builtin_amdgcn_s_sleep(4);
                            return;
                        }
                    }
                }
                BAR(); ++t;
                if (t == 3 && lane == 0) __hip_atomic_store(cnt, call, __ATOMIC_RELAXED, __HIP_MEMORY_SCOPE_AGENT);
            };
            while (__builtin_amdgcn_s_memrealtime() < tgo) attend();
            char* dws = P.ws[dest];
            const __amdgpu_buffer_rsrc_t rs = MKRS(dws + WS_DATA + (size_t)((call & 1u) * NR + rank) * SRC_BYTES + (size_t)stage * UNIT_BYTES, UNIT_BYTES);
#pragma unroll
            for (int i = 0; i < 16; ++i)
                __builtin_amdgcn_raw_buffer_store_b128(v[i], rs, stage_off(quarter * 128 + 8 * i + (lane >> 3)) + (lane & 7) * 16, 0, 17);
            while (vm_outstanding() != 0u) attend();
            WAIT_V(0);
            unsigned* fl = (unsigned*)(dws + WS_FLAG) + (rank * 8 + stage) * 4 + quarter;
            if (lane < NREP) __hip_atomic_store(fl + lane * 256, call, __ATOMIC_RELAXED, __HIP_MEMORY_SCOPE_SYSTEM);
        }
        if (t < 3) {
            while (t < 3) { BAR(); ++t; }
            if (lane == 0) __hip_atomic_store(cnt, call, __ATOMIC_RELAXED, __HIP_MEMORY_SCOPE_AGENT);
        }
    }
}

extern "C" void kernel_launch(int rank, int nranks, ncclComm_t comm,
                              hipStream_t stream,
                              void* const* d_in, const int* in_sizes, int n_in,
                              void* d_out, int out_size,
                              const int* in_dtypes, int out_dtype,
                              void* const* d_out_all,
                              void* const* d_ws_all, size_t ws_bytes) {
    using namespace cfg;
    (void)comm; (void)in_dtypes; (void)out_dtype; (void)d_out_all;
    if (nranks != NR || n_in != 2 || (unsigned)rank >= (unsigned)NR || ws_bytes < WS_NEED ||
        in_sizes[0] != KT * KB || in_sizes[1] != KT * NT || out_size != ML * NT) {
        fprintf(stderr, "kernel_launch: unexpected problem shape\n");
        return;
    }
    static bool attr_set = false;
    if (!attr_set) {
        HIP_CHECK(hipFuncSetAttribute((const void*)a2a_gemm_kernel, hipFuncAttributeMaxDynamicSharedMemorySize, LDS_BYTES));
        attr_set = true;
    }
    Params P;
    P.x = (const char*)d_in[0];
    P.w = (const char*)d_in[1];
    P.out = (char*)d_out;
    for (int p = 0; p < NR; ++p) P.ws[p] = (char*)d_ws_all[p];
    P.rank = rank;
    P.mode = 0;
    P.dbg = nullptr;
    hipLaunchKernelGGL(a2a_gemm_kernel, dim3(NWG), dim3(512), LDS_BYTES, stream, P);
    HIP_CHECK(hipGetLastError());
}
```
